```python
import numpy as np

kernel_name = "gpu_gemm_a2a_m4096_k4096_n4096_bf16_silu"


ABSMAX_RTOL = 0.008

COMPUTE_DTYPE = "bfloat16"
EPILOGUE = "silu"
IN_QUANT = None
OUT_QUANT = None
IN_DTYPES = {"x": "bfloat16", "w_mat": "bfloat16"}
WORLD_SIZE = 8

def _to_bf16(x):
    u = np.ascontiguousarray(x, dtype=np.float32).view(np.uint32)
    return ((u + 0x7FFF + ((u >> 16) & 1)) & 0xFFFF0000).view(np.float32)

def setup_inputs():

    x = _to_bf16(np.random.randn(4096, 4096).astype(np.float32))
    w_mat = _to_bf16((np.random.randn(4096, 4096) / np.sqrt(4096)).astype(np.float32))
    return {"x": x, "w_mat": w_mat}

def _epilogue(y):
    return (y / (1.0 + np.exp(-np.clip(y, -60.0, 60.0)))).astype(np.float32)

def reference(x, w_mat):
    y = (x @ w_mat).astype(np.float32)
    return _epilogue(y)
```

```cpp
#include <hip/hip_runtime.h>
#include <rccl/rccl.h>
#include <cstdio>

#define HIP_CHECK(expr) do { hipError_t _e = (expr); if (_e != hipSuccess) { \
    fprintf(stderr, "HIP error %s at %s:%d\n", hipGetErrorString(_e), __FILE__, __LINE__); \
    return; } } while (0)

namespace {

#define LAS __attribute__((address_space(3)))
#define GAS __attribute__((address_space(1)))
typedef short bf16x8 __attribute__((ext_vector_type(8)));
typedef short bf16x4 __attribute__((ext_vector_type(4)));
typedef float f32x4 __attribute__((ext_vector_type(4)));
typedef unsigned u32x4 __attribute__((ext_vector_type(4)));
typedef unsigned u32x2 __attribute__((ext_vector_type(2)));

constexpr int NR = 8;
constexpr int MLOC = 512;
constexpr int KD = 4096, ND = 4096;
constexpr int NCOL = ND / NR;
constexpr int BM = 128, BN = 64, BK = 64;
constexpr int NT = KD / BK;
constexpr int A_BYTES = BM * BK * 2;
constexpr int B_BYTES = BK * BN * 2;
constexpr int STAGE = A_BYTES + B_BYTES;
#define NSTAGE 6
constexpr int NS = NSTAGE;
constexpr int LDS_BYTES = NS * STAGE;
constexpr int NWG = 256;
constexpr int TILES_PER_DST = NWG / NR;
constexpr int LPT = 6;

constexpr size_t WS_FLAGS = 0;
constexpr size_t WS_FLAGS_BYTES = (size_t)NR * TILES_PER_DST * 128;
constexpr size_t WS_CNT = WS_FLAGS + WS_FLAGS_BYTES;
constexpr size_t WS_CNT_BYTES = (size_t)NWG * 128;
constexpr size_t WS_TMO = WS_CNT + WS_CNT_BYTES;
constexpr size_t WS_STAMP = WS_TMO + 128;
constexpr size_t WS_END = WS_STAMP + (size_t)NWG * 128;
constexpr unsigned long long DEADLINE_TICKS = 20000000ull;

struct Params {
    const char* x;
    const char* w;
    char* const* out_all;
    char* const* ws_all;
    int rank;
    int pad;
};

template <int N> __device__ __forceinline__ void wait_vm() { asm volatile("s_waitcnt vmcnt(%0)" :: "n"(N) : "memory"); }
__device__ __forceinline__ void bar() { asm volatile("" ::: "memory"); __builtin_amdgcn_s_barrier(); asm volatile("" ::: "memory"); }

__device__ __forceinline__ float silu(float y) {
    const float c = __builtin_amdgcn_fmed3f(y, -60.0f, 60.0f);
    const float e = __builtin_amdgcn_exp2f(c * -1.44269504088896340736f);
    return y * __builtin_amdgcn_rcpf(1.0f + e);
}
__device__ __forceinline__ unsigned cvt_pk(float lo, float hi) {
    unsigned r; asm("v_cvt_pk_bf16_f32 %0, %1, %2" : "=v"(r) : "v"(lo), "v"(hi)); return r;
}

struct Frags { bf16x8 a[2][4]; bf16x4 b[2][2][2]; };

__global__ __launch_bounds__(512, 2) void fused_kernel(Params p) {
    extern __shared__ __attribute__((aligned(16))) unsigned char shm[];
    LAS unsigned char* lds = (LAS unsigned char*)shm;
    const int tid = threadIdx.x;
    const int wid = __builtin_amdgcn_readfirstlane(tid >> 6);
    const int lane = tid & 63;
    const int b = blockIdx.x;
    const int dst = b & 7;
    const int idx = b >> 3;
    const int mi = idx & 3, ni = idx >> 2;
    const int m0 = mi * BM;
    const int n0 = dst * NCOL + ni * BN;

    if (wid >= 4) {
        const int w = wid - 4;
        unsigned offA[4], offB[2];
#pragma unroll
        for (int i = 0; i < 4; ++i) {
            const int row = (w * 4 + i) * 8 + (lane >> 3);
            const int ch = (lane & 7) ^ ((row >> 1) & 7);
            offA[i] = (unsigned)(row * (KD * 2) + ch * 16);
        }
#pragma unroll
        for (int i = 0; i < 2; ++i) {
            const int k = (w * 2 + i) * 8 + (lane >> 3);
            const int s = ((k >> 1) & 1) | (((k >> 3) & 1) << 1);
            const int ch = (lane & 7) ^ (s << 1);
            offB[i] = (unsigned)(k * (ND * 2) + ch * 16);
        }
        const char* gA = p.x + (size_t)m0 * (KD * 2);
        const char* gB = p.w + (size_t)n0 * 2;
        auto issue = [&](int t, int slot) {
            const char* a = gA + (size_t)t * (BK * 2);
            const char* bb = gB + (size_t)t * ((size_t)BK * ND * 2);
            LAS unsigned char* sa = lds + slot * STAGE + w * 4096;
            LAS unsigned char* sb = lds + slot * STAGE + A_BYTES + w * 2048;
#pragma unroll
            for (int i = 0; i < 4; ++i)
                __builtin_amdgcn_global_load_lds((const GAS unsigned*)(a + offA[i]), (LAS unsigned*)(sa + i * 1024), 16, 0, 0);
#pragma unroll
            for (int i = 0; i < 2; ++i)
                __builtin_amdgcn_global_load_lds((const GAS unsigned*)(bb + offB[i]), (LAS unsigned*)(sb + i * 1024), 16, 0, 0);
        };
#pragma unroll
        for (int t = 0; t < NS; ++t) issue(t, t);
        wait_vm<LPT*(NS - 1)>(); bar();
        int slot = 0;
        for (int t = 0; t < NT - NS; ++t) {
            wait_vm<LPT*(NS - 2)>(); bar();
            issue(t + NS, slot);
            slot = (slot + 1 == NS) ? 0 : slot + 1;
        }
#define TAIL(j) if constexpr (NS > (j)) { wait_vm<LPT * ((NS - (j) - 2) > 0 ? (NS - (j) - 2) : 0)>(); bar(); }
        TAIL(0) TAIL(1) TAIL(2) TAIL(3) TAIL(4) TAIL(5) TAIL(6) TAIL(7)
#undef TAIL
    } else {
        const int wr = wid >> 1, wc = wid & 1;
        const int fr = lane & 15, fq = lane >> 4;
        const int arow = wr * 64 + fr;
        const int asw = (arow >> 1) & 7;
        const unsigned aoff0 = arow * 128 + ((fq ^ asw) << 4);
        const unsigned aoff1 = arow * 128 + (((4 + fq) ^ asw) << 4);
        const int q = (lane & 15) >> 2, pp = lane & 3;
        const int sB = ((q >> 1) & 1) | ((fq & 1) << 1);
        unsigned boff[2];
#pragma unroll
        for (int nt = 0; nt < 2; ++nt) boff[nt] = A_BYTES + (8 * fq + q) * 128 + (((wc * 2 + nt) ^ sB) << 5) + 8 * pp;
        f32x4 acc[4][2];
#pragma unroll
        for (int mt = 0; mt < 4; ++mt)
#pragma unroll
            for (int nt = 0; nt < 2; ++nt) acc[mt][nt] = (f32x4){0.f, 0.f, 0.f, 0.f};
        auto rd = [&](Frags& f, int slot) {
            LAS unsigned char* s = lds + slot * STAGE;
#pragma unroll
            for (int ks = 0; ks < 2; ++ks) {
#pragma unroll
                for (int nt = 0; nt < 2; ++nt)
#pragma unroll
                    for (int h = 0; h < 2; ++h)
                        f.b[ks][nt][h] = __builtin_amdgcn_ds_read_tr16_b64_v4i16((LAS bf16x4*)(s + boff[nt] + ks * 4096 + h * 512));
#pragma unroll
                for (int mt = 0; mt < 4; ++mt)
                    f.a[ks][mt] = *(LAS bf16x8*)(s + (ks ? aoff1 : aoff0) + mt * 2048);
            }
        };
        auto mm = [&](const Frags& f) {
#pragma unroll
            for (int ks = 0; ks < 2; ++ks)
#pragma unroll
                for (int nt = 0; nt < 2; ++nt) {
                    const bf16x8 wv = __builtin_shufflevector(f.b[ks][nt][0], f.b[ks][nt][1], 0, 1, 2, 3, 4, 5, 6, 7);
#pragma unroll
                    for (int mt = 0; mt < 4; ++mt)
                        acc[mt][nt] = __builtin_amdgcn_mfma_f32_16x16x32_bf16(wv, f.a[ks][mt], acc[mt][nt], 0, 0, 0);
                }
        };
        Frags f0, f1;
        bar();
        rd(f0, 0);
        int slot = 1;
        for (int t = 0; t < NT; t += 2) {
            asm volatile("s_waitcnt lgkmcnt(0)" ::: "memory");
            bar();
            rd(f1, slot); slot = (slot + 1 == NS) ? 0 : slot + 1;
            mm(f0);
            asm volatile("s_waitcnt lgkmcnt(0)" ::: "memory");
            bar();
            if (t + 2 < NT) { rd(f0, slot); slot = (slot + 1 == NS) ? 0 : slot + 1; }
            mm(f1);
        }
#pragma unroll
        for (int mt = 0; mt < 4; ++mt)
#pragma unroll
            for (int nt = 0; nt < 2; ++nt) {
                const f32x4 v = acc[mt][nt];
                u32x2 o; o.x = cvt_pk(silu(v[0]), silu(v[1])); o.y = cvt_pk(silu(v[2]), silu(v[3]));
                const int row = wr * 64 + mt * 16 + fr;
                const int col = wc * 32 + nt * 16 + fq * 4;
                const int ch = (col >> 3) ^ (row & 7);
                *(LAS u32x2*)(lds + row * 128 + ch * 16 + (col & 4) * 2) = o;
            }
        asm volatile("s_waitcnt lgkmcnt(0)" ::: "memory");
    }
    bar();
    {
        char* obase = p.out_all[dst] + ((size_t)(p.rank * MLOC + m0) * NCOL + ni * BN) * 2;
#pragma unroll
        for (int i = 0; i < 2; ++i) {
            const int row = (wid * 2 + i) * 8 + (lane >> 3);
            const int ch = lane & 7;
            const u32x4 v = *(LAS u32x4*)(lds + row * 128 + ((ch ^ (row & 7)) << 4));
            char* ptr = obase + (size_t)row * (NCOL * 2) + ch * 16;
            asm volatile("global_store_dwordx4 %0, %1, off sc0 sc1\n\ts_nop 1" :: "v"(ptr), "v"(v) : "memory");
        }
    }
    asm volatile("s_waitcnt vmcnt(0)" ::: "memory");
    bar();
    char* myws = p.ws_all[p.rank];
    const int src = p.rank;
    unsigned call = 0;
    if (wid == 0) {
        unsigned* cnt = (unsigned*)(myws + WS_CNT + (size_t)b * 128);
        call = __hip_atomic_load(cnt, __ATOMIC_RELAXED, __HIP_MEMORY_SCOPE_AGENT) + 1;
        if (lane == 0) {
            __hip_atomic_store(cnt, call, __ATOMIC_RELAXED, __HIP_MEMORY_SCOPE_AGENT);
            unsigned* flag = (unsigned*)(p.ws_all[dst] + WS_FLAGS + (size_t)(src * TILES_PER_DST + idx) * 128);
            __hip_atomic_store(flag, call, __ATOMIC_RELAXED, __HIP_MEMORY_SCOPE_SYSTEM);
        }
    }
    if (b == 0) {
        if (wid == 0 && lane == 0) *(LAS unsigned*)(lds + 32768) = call;
        asm volatile("s_waitcnt lgkmcnt(0)" ::: "memory");
        bar();
        if (wid < 4) {
            const unsigned c = *(LAS unsigned*)(lds + 32768);
            unsigned* flag = (unsigned*)(myws + WS_FLAGS + (size_t)tid * 128);
            unsigned* tmo = (unsigned*)(myws + WS_TMO);
            const unsigned long long t0 = __builtin_amdgcn_s_memrealtime();
            bool timed_out = false;
            while (true) {
                const unsigned v = __hip_atomic_load(flag, __ATOMIC_RELAXED, __HIP_MEMORY_SCOPE_SYSTEM);
                if (__all((int)(v - c) >= 0)) break;
                if (__hip_atomic_load(tmo, __ATOMIC_RELAXED, __HIP_MEMORY_SCOPE_AGENT) == c
                    || __builtin_amdgcn_s_memrealtime() - t0 > DEADLINE_TICKS) {
                    __hip_atomic_store(tmo, c, __ATOMIC_RELAXED, __HIP_MEMORY_SCOPE_AGENT);
                    timed_out = true;
                    break;
                }
                __builtin_amdgcn_s_sleep(2);
            }
            if (timed_out && lane == 0) {
                unsigned short* o = (unsigned short*)(p.out_all[p.rank] + (size_t)p.rank * MLOC * NCOL * 2);
                __hip_atomic_store(o, (unsigned short)0x7fc0, __ATOMIC_RELAXED, __HIP_MEMORY_SCOPE_SYSTEM);
            }
        }
    }
}

}

extern "C" void kernel_launch(int rank, int nranks, ncclComm_t comm, hipStream_t stream,
                              void* const* d_in, const int* in_sizes, int n_in,
                              void* d_out, int out_size, const int* in_dtypes, int out_dtype,
                              void* const* d_out_all, void* const* d_ws_all, size_t ws_bytes) {
    (void)comm; (void)in_sizes; (void)n_in; (void)d_out; (void)out_size; (void)in_dtypes; (void)out_dtype;
    if (nranks != NR || ws_bytes < WS_END) { fprintf(stderr, "unsupported configuration\n"); return; }
    static bool attr = false;
    if (!attr) {
        HIP_CHECK(hipFuncSetAttribute((const void*)fused_kernel, hipFuncAttributeMaxDynamicSharedMemorySize, LDS_BYTES));
        attr = true;
    }
    Params p;
    p.x = (const char*)d_in[0];
    p.w = (const char*)d_in[1];
    p.out_all = (char* const*)d_out_all;
    p.ws_all = (char* const*)d_ws_all;
    p.rank = rank;
    p.pad = 0;
    hipLaunchKernelGGL(fused_kernel, dim3(NWG), dim3(512), LDS_BYTES, stream, p);
    HIP_CHECK(hipGetLastError());
}
```

```cpp
#include <hip/hip_runtime.h>
#include <rccl/rccl.h>
#include <cstdio>

#define HIP_CHECK(expr) do { hipError_t _e = (expr); if (_e != hipSuccess) { \
    fprintf(stderr, "HIP error %s at %s:%d\n", hipGetErrorString(_e), __FILE__, __LINE__); \
    return; } } while (0)

namespace {

#define LAS __attribute__((address_space(3)))
#define GAS __attribute__((address_space(1)))
typedef short bf16x8 __attribute__((ext_vector_type(8)));
typedef short bf16x4 __attribute__((ext_vector_type(4)));
typedef float f32x4 __attribute__((ext_vector_type(4)));
typedef float f32x16 __attribute__((ext_vector_type(16)));
typedef unsigned u32x4 __attribute__((ext_vector_type(4)));
typedef unsigned u32x2 __attribute__((ext_vector_type(2)));
typedef GAS unsigned gu32;
typedef GAS unsigned short gu16;

constexpr int NR = 8;
constexpr int MLOC = 512;
constexpr int KD = 4096, ND = 4096;
constexpr int NCOL = ND / NR;
constexpr int BM = 128, BN = 128, BK = 64;
constexpr int SPP = 16;
constexpr int NSTEP = 2 * SPP;
constexpr int SLOT = 16384;
constexpr int NS = 4;
constexpr int LDS_B = NS * SLOT;
constexpr int LDS_STG = 2 * NS * SLOT;
constexpr int LDS_BYTES = LDS_STG + 8192;
constexpr int NWG = 256;
constexpr int LPT = 8;
#define LEAD_BS 1

constexpr size_t WS_FLAGS = 0;
constexpr size_t WS_CNT = WS_FLAGS + (size_t)NWG * 128;
constexpr size_t WS_TMO = WS_CNT + (size_t)NWG * 128;
constexpr size_t WS_STAMP = WS_TMO + 128;
constexpr size_t WS_XFLAG = WS_STAMP + (size_t)NWG * 128;
constexpr size_t WS_XFLAG_BYTES = (size_t)2 * 64 * 4 * 4 * 128;
constexpr size_t WS_INBOX = 1u << 20;
constexpr size_t INBOX_TILE = (size_t)4 * 3 * 4 * 4 * 1024;
constexpr size_t WS_INBOX_BYTES = 2 * 64 * INBOX_TILE;
constexpr size_t WS_END = WS_INBOX + WS_INBOX_BYTES;
static_assert(WS_XFLAG + WS_XFLAG_BYTES <= WS_INBOX, "layout");
constexpr unsigned long long DEADLINE_TICKS = 20000000ull;

struct Params {
    const char* x;
    const char* w;
    char* const* out_all;
    char* const* ws_all;
    int rank;
    int pad;
};

template <int N> __device__ __forceinline__ void wait_vm() { asm volatile("s_waitcnt vmcnt(%0)" :: "n"(N) : "memory"); }
__device__ __forceinline__ void bar() { asm volatile("" ::: "memory"); __builtin_amdgcn_s_barrier(); asm volatile("" ::: "memory"); }
#define SCHED() __builtin_amdgcn_sched_barrier(0)

__device__ __forceinline__ float silu(float y) {
    const float c = __builtin_amdgcn_fmed3f(y, -60.0f, 60.0f);
    const float e = __builtin_amdgcn_exp2f(c * -1.44269504088896340736f);
    return y * __builtin_amdgcn_rcpf(1.0f + e);
}
__device__ __forceinline__ unsigned cvt_pk(float lo, float hi) {
    unsigned r; asm("v_cvt_pk_bf16_f32 %0, %1, %2" : "=v"(r) : "v"(lo), "v"(hi)); return r;
}
__device__ __forceinline__ __amdgpu_buffer_rsrc_t make_rsrc(const char* base, unsigned bytes) {
    const unsigned long long a = (unsigned long long)base;
    const unsigned lo = __builtin_amdgcn_readfirstlane((unsigned)a), hi = __builtin_amdgcn_readfirstlane((unsigned)(a >> 32));
    return __builtin_amdgcn_make_buffer_rsrc((void*)(((unsigned long long)hi << 32) | lo), 0, bytes, 0x00020000);
}
constexpr int AUX_SYS = 17;

struct F { bf16x8 a[2][4]; bf16x4 b[2][2]; };

__device__ __forceinline__ int korder(int tt, int g) {
    constexpr int bs = LEAD_BS, nb = SPP / bs;
    const int tau = tt / bs, s = tt % bs;
    int u = tau <= g ? tau : tau + 3 - g;
    if (u >= nb) u += g + 1;
    return (g + 4 * (u >> 2) - (u & 3)) * bs + s;
}

__global__ __launch_bounds__(512, 2) void fused_kernel(Params p) {
    extern __shared__ __attribute__((aligned(16))) unsigned char shm[];
    LAS unsigned char* lds = (LAS unsigned char*)shm;
    const int tid = threadIdx.x;
    const int wid = __builtin_amdgcn_readfirstlane(tid >> 6);
    const int lane = tid & 63;
    const int b = blockIdx.x;
    const int kq = b & 3, hh = (b >> 2) & 1, idx = b >> 3;
    const int mi = idx & 3, nl = idx >> 2;
    const int tile = hh * 8 + nl;
    const int dst = tile >> 1, sub = tile & 1;
    const int T = hh * 32 + idx;
    const int m0 = mi * BM;
    char* myws = p.ws_all[p.rank];
    unsigned call = 0;

    if (wid >= 4) {
        const int w = wid - 4;
        unsigned offA[4], offB[4];
#pragma unroll
        for (int i = 0; i < 4; ++i) {
            const int row = (w * 4 + i) * 8 + (lane >> 3);
            const int ch = (lane & 7) ^ ((row >> 1) & 7);
            offA[i] = (unsigned)(row * (KD * 2) + ch * 16);
            const int k = (w * 4 + i) * 4 + (lane >> 4);
            const int cb = (lane & 15) ^ ((k & 3) << 2);
            offB[i] = (unsigned)(k * (ND * 2) + cb * 16);
        }
        const char* gA = p.x + (size_t)m0 * (KD * 2) + (size_t)kq * (SPP * BK * 2);
        const char* gB = p.w + (size_t)kq * ((size_t)SPP * BK * ND * 2) + (size_t)(dst * NCOL + sub * BN) * 2;
        const int g = (mi + nl) & 3;
        const unsigned ldsb = (unsigned)(size_t)lds;
        struct Src { const char* a; const char* b; };
        auto source = [&](int t) -> Src {
            const int kt = korder(t & (SPP - 1), g);
            return Src{gA + kt * (BK * 2), gB + (size_t)kt * ((size_t)BK * ND * 2) + (t >> 4) * (256 * 2)};
        };
        auto issue = [&](const Src& q, int slot) {
            const unsigned la = ldsb + slot * SLOT + w * 4096;
#pragma unroll
            for (int i = 0; i < 4; ++i)
                asm volatile("s_mov_b32 m0, %2\n\ts_nop 0\n\tglobal_load_lds_dwordx4 %0, %1" :: "v"(offA[i]), "s"(q.a), "s"(la + i * 1024) : "memory");
#pragma unroll
            for (int i = 0; i < 4; ++i)
                asm volatile("s_mov_b32 m0, %2\n\ts_nop 0\n\tglobal_load_lds_dwordx4 %0, %1" :: "v"(offB[i]), "s"(q.b), "s"(la + LDS_B + i * 1024) : "memory");
        };
        issue(source(0), 0);
        issue(source(1), 1);
        Src nx = source(2);
#pragma unroll 1
        for (int t = -1; t <= NSTEP - 4; ++t) {
            wait_vm<LPT>(); bar();
            issue(nx, (t + 3) & (NS - 1));
            nx = source((t + 4) & (NSTEP - 1));
        }
        wait_vm<LPT>(); bar();
        wait_vm<0>(); bar();
        bar();
        bar();
    } else {
        __builtin_amdgcn_s_setprio(2);
        const int w = wid;
        const int l31 = lane & 31, h = lane >> 5;
        bool bad = false;
        {
            const gu32* cnt = (const gu32*)(myws + WS_CNT + (size_t)b * 128);
            call = __hip_atomic_load(cnt, __ATOMIC_RELAXED, __HIP_MEMORY_SCOPE_AGENT) + 1;
        }
        const unsigned ldsb = (unsigned)(size_t)lds;
        unsigned xa[4][4];
#pragma unroll
        for (int j = 0; j < 4; ++j)
#pragma unroll
            for (int ks = 0; ks < 4; ++ks) xa[j][ks] = ldsb + ((j ^ kq) * 32 + l31) * 128 + (((2 * ks + h) ^ ((l31 >> 1) & 7)) << 4);
        const int blk = (lane >> 4) & 1, q4 = (lane & 15) >> 2, p4 = lane & 3;
        unsigned wa = ldsb + LDS_B + (8 * h + q4) * 256 + (((2 * w + blk) ^ (q4 << 1)) << 5) + 8 * p4;
        f32x16 acc[4];
#pragma unroll
        for (int j = 0; j < 4; ++j)
#pragma unroll
            for (int e = 0; e < 16; ++e) acc[j][e] = 0.f;
        auto rd1 = [&](F& f, int so, int hf, int r) {
            const int s = r / 6, rr = r % 6, ks = 2 * hf + s;
            if (rr < 2) f.b[s][rr] = __builtin_amdgcn_ds_read_tr16_b64_v4i16((LAS bf16x4*)(size_t)(wa + so + ks * 4096 + rr * 1024));
            else f.a[s][rr - 2] = *(LAS bf16x8*)(size_t)(xa[rr - 2][ks] + so);
        };
        auto half = [&](const F& cur, F& nxt, int so, int hf) {
#pragma unroll
            for (int s = 0; s < 2; ++s) {
                const bf16x8 wv = __builtin_shufflevector(cur.b[s][0], cur.b[s][1], 0, 1, 2, 3, 4, 5, 6, 7);
#pragma unroll
                for (int j = 0; j < 4; ++j) {
                    asm volatile("v_mfma_f32_32x32x16_bf16 %0, %1, %2, %0" : "+v"(acc[j]) : "v"(wv), "v"(cur.a[s][j]));
                    SCHED();
                    const int i = s * 4 + j;
                    const int r0 = i < 4 ? 2 * i : 4 + i, n = i < 4 ? 2 : 1;
                    rd1(nxt, so, hf, r0); if (n == 2) rd1(nxt, so, hf, r0 + 1);
                    SCHED();
                }
            }
        };
        auto toggle = [&]() {
#pragma unroll
            for (int j = 0; j < 4; ++j)
#pragma unroll
                for (int ks = 0; ks < 4; ++ks) xa[j][ks] ^= 2 * SLOT;
            wa ^= 2 * SLOT;
        };
        auto mfma_done = [&]() { SCHED(); asm volatile("s_nop 7\n\ts_nop 7" ::: "memory"); SCHED(); };
        const __amdgpu_buffer_rsrc_t rs_ws = make_rsrc(myws, (unsigned)WS_END);
        const __amdgpu_buffer_rsrc_t rs_out = make_rsrc(p.out_all[dst], (unsigned)(ND * NCOL * 2));
        const unsigned l16 = lane * 16;
        auto inbox = [&](int ph, int q, int j) -> unsigned {
            return (unsigned)(WS_INBOX + (size_t)(ph * 64 + T) * INBOX_TILE + (size_t)((q * 3 + (j - 1)) * 4 + w) * 4096);
        };
        auto xflag = [&](int ph, int q) -> unsigned { return (unsigned)(WS_XFLAG + (size_t)(((ph * 64 + T) * 4 + q) * 4 + w) * 128); };
        auto quad = [&](const f32x16& v, int e4) -> f32x4 { return (f32x4){v[4 * e4], v[4 * e4 + 1], v[4 * e4 + 2], v[4 * e4 + 3]}; };
        auto send_partials = [&](int ph) {
#pragma unroll
            for (int j = 1; j < 4; ++j) {
                const unsigned o = inbox(ph, kq ^ j, j);
#pragma unroll
                for (int e4 = 0; e4 < 4; ++e4)
                    __builtin_amdgcn_raw_buffer_store_b128(__builtin_bit_cast(u32x4, quad(acc[j], e4)), rs_ws, l16 + e4 * 1024, o, AUX_SYS);
            }
        };
        auto publish_partials = [&](int ph) {
            wait_vm<0>();
            if (lane >= 1 && lane < 4) {
                gu32* f = (gu32*)(myws + xflag(ph, kq ^ lane) + (lane - 1) * 4);
                __hip_atomic_store(f, call, __ATOMIC_RELAXED, __HIP_MEMORY_SCOPE_SYSTEM);
            }
        };
        auto wait_partials = [&](int ph) {
            gu32* f = (gu32*)(myws + xflag(ph, kq) + (lane < 3 ? lane : 0) * 4);
            gu32* tmo = (gu32*)(myws + WS_TMO);
            unsigned v = __hip_atomic_load(f, __ATOMIC_RELAXED, __HIP_MEMORY_SCOPE_SYSTEM);
            if (!__all((int)(v - call) >= 0)) {
                const unsigned long long t0 = __builtin_amdgcn_s_memrealtime();
                while (true) {
                    __builtin_amdgcn_s_sleep(1);
                    v = __hip_atomic_load(f, __ATOMIC_RELAXED, __HIP_MEMORY_SCOPE_SYSTEM);
                    if (__all((int)(v - call) >= 0)) break;
                    if (__hip_atomic_load(tmo, __ATOMIC_RELAXED, __HIP_MEMORY_SCOPE_AGENT) == call
                        || __builtin_amdgcn_s_memrealtime() - t0 > DEADLINE_TICKS) {
                        __hip_atomic_store(tmo, call, __ATOMIC_RELAXED, __HIP_MEMORY_SCOPE_AGENT);
                        bad = true;
                        break;
                    }
                }
            }
            asm volatile("" ::: "memory");
        };
        auto load_partials = [&](int ph, f32x4 (&in)[3][4]) {
#pragma unroll
            for (int j = 1; j < 4; ++j) {
                const unsigned o = inbox(ph, kq, j);
#pragma unroll
                for (int e4 = 0; e4 < 4; ++e4)
                    in[j - 1][e4] = __builtin_bit_cast(f32x4, __builtin_amdgcn_raw_buffer_load_b128(rs_ws, l16 + e4 * 1024, o, AUX_SYS));
            }
        };
        auto finish_a = [&](const f32x16& own, const f32x4 (&in)[3][4]) {
            LAS unsigned char* stg = lds + LDS_STG + l31 * 256 + h * 8;
#pragma unroll
            for (int e4 = 0; e4 < 4; ++e4) {
                const f32x4 v = ((quad(own, e4) + in[0][e4]) + in[1][e4]) + in[2][e4];
                u32x2 o; o.x = cvt_pk(silu(v[0]), silu(v[1])); o.y = cvt_pk(silu(v[2]), silu(v[3]));
                if (bad) { o.x = 0x7fc07fc0u; o.y = 0x7fc07fc0u; }
                *(LAS u32x2*)(stg + (((w * 4 + e4) ^ (l31 & 15)) << 4)) = o;
            }
            asm volatile("s_waitcnt lgkmcnt(0)" ::: "memory");
        };
        auto finish_b = [&](int ph) {
            const unsigned obase = (unsigned)(((p.rank * MLOC + m0 + kq * 32) * NCOL + ph * 256 + sub * 128) * 2);
#pragma unroll
            for (int i = 0; i < 2; ++i) {
                const int r = (w * 2 + i) * 4 + (lane >> 4), c = lane & 15;
                const u32x4 v = *(LAS u32x4*)(lds + LDS_STG + r * 256 + ((c ^ (r & 15)) << 4));
                __builtin_amdgcn_raw_buffer_store_b128(v, rs_out, r * (NCOL * 2) + c * 16, obase, AUX_SYS);
            }
        };

        F f0, f1;
        f32x16 keep;
        f32x4 in[3][4];
        auto steps2 = [&](int npairs) {
#pragma unroll 1
            for (int u = 0; u < npairs; ++u) {
                bar();
                half(f0, f1, 0, 1);
                half(f1, f0, SLOT, 0);
                bar();
                half(f0, f1, SLOT, 1);
                toggle();
                SCHED();
                half(f1, f0, 0, 0);
            }
        };
        bar();
#pragma unroll
        for (int r = 0; r < 12; ++r) rd1(f0, 0, 0, r);
        asm volatile("s_nop 1" ::: "memory");
        steps2(SPP / 2);
        mfma_done();
        send_partials(0);
        keep = acc[0];
#pragma unroll
        for (int j = 0; j < 4; ++j)
#pragma unroll
            for (int e = 0; e < 16; ++e) acc[j][e] = 0.f;
        SCHED();
        asm volatile("s_nop 1" ::: "memory");
        steps2(2);
        publish_partials(0);
        steps2(2);
        wait_partials(0);
        load_partials(0, in);
        steps2(2);
        finish_a(keep, in);
        steps2(1);
        finish_b(0);
        steps2(1);
        mfma_done();
        send_partials(1);
        publish_partials(1);
        wait_partials(1);
        load_partials(1, in);
        finish_a(acc[0], in);
        bar();
        finish_b(1);
        wait_vm<0>();
    }
    bar();
    if (wid == 0) {
        const int src = p.rank;
        gu32* cnt = (gu32*)(myws + WS_CNT + (size_t)b * 128);
        if (lane == 0) {
            gu32* flag = (gu32*)(p.ws_all[dst] + WS_FLAGS + (size_t)(src * 32 + kq * 8 + mi * 2 + sub) * 128);
            __hip_atomic_store(flag, call, __ATOMIC_RELAXED, __HIP_MEMORY_SCOPE_SYSTEM);
            __hip_atomic_store(cnt, call, __ATOMIC_RELAXED, __HIP_MEMORY_SCOPE_AGENT);
            gu32* mine = (gu32*)(myws + WS_FLAGS + (size_t)b * 128);
            gu32* tmo = (gu32*)(myws + WS_TMO);
            const unsigned long long t0 = __builtin_amdgcn_s_memrealtime();
            while ((int)(__hip_atomic_load(mine, __ATOMIC_RELAXED, __HIP_MEMORY_SCOPE_SYSTEM) - call) < 0) {
                if (__hip_atomic_load(tmo, __ATOMIC_RELAXED, __HIP_MEMORY_SCOPE_AGENT) == call
                    || __builtin_amdgcn_s_memrealtime() - t0 > DEADLINE_TICKS) {
                    __hip_atomic_store(tmo, call, __ATOMIC_RELAXED, __HIP_MEMORY_SCOPE_AGENT);
                    gu16* o = (gu16*)(p.out_all[p.rank] + (size_t)p.rank * MLOC * NCOL * 2);
                    __hip_atomic_store(o, (unsigned short)0x7fc0, __ATOMIC_RELAXED, __HIP_MEMORY_SCOPE_SYSTEM);
                    break;
                }
                __builtin_amdgcn_s_sleep(1);
            }
        }
    }
}

}

extern "C" void kernel_launch(int rank, int nranks, ncclComm_t comm, hipStream_t stream,
                              void* const* d_in, const int* in_sizes, int n_in,
                              void* d_out, int out_size, const int* in_dtypes, int out_dtype,
                              void* const* d_out_all, void* const* d_ws_all, size_t ws_bytes) {
    (void)comm; (void)in_sizes; (void)n_in; (void)d_out; (void)out_size; (void)in_dtypes; (void)out_dtype;
    if (nranks != NR || ws_bytes < WS_END) { fprintf(stderr, "unsupported configuration\n"); return; }
    static bool attr = false;
    if (!attr) {
        HIP_CHECK(hipFuncSetAttribute((const void*)fused_kernel, hipFuncAttributeMaxDynamicSharedMemorySize, LDS_BYTES));
        attr = true;
    }
    Params p;
    p.x = (const char*)d_in[0];
    p.w = (const char*)d_in[1];
    p.out_all = (char* const*)d_out_all;
    p.ws_all = (char* const*)d_ws_all;
    p.rank = rank;
    p.pad = 0;
    hipLaunchKernelGGL(fused_kernel, dim3(NWG), dim3(512), LDS_BYTES, stream, p);
    HIP_CHECK(hipGetLastError());
}
```

```cpp
#include <hip/hip_runtime.h>
#include <rccl/rccl.h>
#include <cstdio>
#include <type_traits>

#define HIP_CHECK(expr) do { hipError_t _e = (expr); if (_e != hipSuccess) { \
    fprintf(stderr, "HIP error %s at %s:%d\n", hipGetErrorString(_e), __FILE__, __LINE__); \
    return; } } while (0)

namespace {

#define LAS __attribute__((address_space(3)))
#define GAS __attribute__((address_space(1)))
typedef short bf16x8 __attribute__((ext_vector_type(8)));
typedef short bf16x4 __attribute__((ext_vector_type(4)));
typedef float f32x4 __attribute__((ext_vector_type(4)));
typedef float f32x16 __attribute__((ext_vector_type(16)));
typedef unsigned u32x4 __attribute__((ext_vector_type(4)));
typedef unsigned u32x2 __attribute__((ext_vector_type(2)));
typedef GAS unsigned gu32;
typedef GAS unsigned short gu16;

constexpr int NR = 8;
constexpr int MLOC = 512;
constexpr int KD = 4096, ND = 4096;
constexpr int NCOL = ND / NR;
constexpr int BM = 128, BN = 128, BK = 64;
constexpr int SPP = 16;
constexpr int NSTEP = 2 * SPP;
constexpr int SLOT = 16384;
constexpr int NS = 4;
constexpr int LDS_B = NS * SLOT;
constexpr int LDS_XB = 2 * NS * SLOT;
constexpr int LDS_STG0 = LDS_XB + 3 * 8192;
constexpr int LDS_OWN = 0;
constexpr int LDS_STG1 = 16384;
constexpr int LDS_BYTES = LDS_STG0 + 8192;
constexpr int NWG = 256;
constexpr int LPT = 8;
#define XT 19
#define NCLS 8
#define TSLEEP 4
#define FBM 3
#define LEAD_BS 1

constexpr size_t WS_FLAGS = 0;
constexpr size_t WS_CNT = WS_FLAGS + (size_t)NWG * 128;
constexpr size_t WS_TMO = WS_CNT + (size_t)NWG * 128;
constexpr size_t WS_STAMP = WS_TMO + 128;
constexpr size_t WS_INBOX = 1u << 20;
constexpr size_t INBOX_TILE = (size_t)4 * 3 * 4 * 2 * 1024;
constexpr size_t WS_INBOX_BYTES = 2 * 64 * INBOX_TILE;
constexpr size_t WS_END = WS_INBOX + WS_INBOX_BYTES;
static_assert(WS_STAMP + (size_t)NWG * 128 <= WS_INBOX, "layout");
constexpr unsigned long long DEADLINE_TICKS = 20000000ull;

struct Params {
    const char* x;
    const char* w;
    char* out[NR];
    char* ws[NR];
    int rank;
    int pad;
};

template <int N> __device__ __forceinline__ void wait_vm() { asm volatile("s_waitcnt vmcnt(%0)" :: "n"(N) : "memory"); }
__device__ __forceinline__ void wait_lds() { asm volatile("s_waitcnt lgkmcnt(0)" ::: "memory"); }
__device__ __forceinline__ void bar() { asm volatile("" ::: "memory"); __builtin_amdgcn_s_barrier(); asm volatile("" ::: "memory"); }
#define SCHED() __builtin_amdgcn_sched_barrier(0)
#define ACLOB0 "a0","a1","a2","a3","a4","a5","a6","a7","a8","a9","a10","a11","a12","a13","a14","a15"
#define ACLOB1 "a16","a17","a18","a19","a20","a21","a22","a23","a24","a25","a26","a27","a28","a29","a30","a31"
#define ACLOB2 "a32","a33","a34","a35","a36","a37","a38","a39","a40","a41","a42","a43","a44","a45","a46","a47"
#define ACLOB3 "a48","a49","a50","a51","a52","a53","a54","a55","a56","a57","a58","a59","a60","a61","a62","a63"
#define ACLOB4 "a64","a65","a66","a67","a68","a69","a70","a71","a72","a73","a74","a75","a76","a77","a78","a79"
#define ACLOB5 "a80","a81","a82","a83","a84","a85","a86","a87","a88","a89","a90","a91","a92","a93","a94","a95"
#define ACLOB6 "a96","a97","a98","a99","a100","a101","a102","a103","a104","a105","a106","a107","a108","a109","a110","a111"
#define ACLOB7 "a112","a113","a114","a115","a116","a117","a118","a119","a120","a121","a122","a123","a124","a125","a126","a127"
template <int R> __device__ __forceinline__ float areg_read() { float v; asm volatile("v_accvgpr_read_b32 %0, a%c1" : "=v"(v) : "i"(R)); return v; }
template <int R> __device__ __forceinline__ void areg_write(float v) { asm volatile("v_accvgpr_write_b32 a%c0, %1" :: "i"(R), "v"(v)); }
template <int B> __device__ __forceinline__ f32x4 areg_quad() { return (f32x4){areg_read<B>(), areg_read<B + 1>(), areg_read<B + 2>(), areg_read<B + 3>()}; }

__device__ __forceinline__ float silu(float y) {
    const float c = __builtin_amdgcn_fmed3f(y, -60.0f, 60.0f);
    const float e = __builtin_amdgcn_exp2f(c * -1.44269504088896340736f);
    return y * __builtin_amdgcn_rcpf(1.0f + e);
}
__device__ __forceinline__ unsigned cvt_pk(float lo, float hi) {
    unsigned r; asm("v_cvt_pk_bf16_f32 %0, %1, %2" : "=v"(r) : "v"(lo), "v"(hi)); return r;
}
__device__ __forceinline__ __amdgpu_buffer_rsrc_t make_rsrc(const char* base, unsigned bytes) {
    return __builtin_amdgcn_make_buffer_rsrc((void*)base, 0, bytes, 0x00020000);
}
__device__ __forceinline__ f32x4 quad(const f32x16& v, int e4) { return (f32x4){v[4 * e4], v[4 * e4 + 1], v[4 * e4 + 2], v[4 * e4 + 3]}; }
constexpr int AUX_SYS = 17;

struct F { bf16x8 a[2][4]; bf16x4 b[2][2]; };

__device__ __forceinline__ int korder(int tt, int g) {
    constexpr int bs = LEAD_BS, nb = SPP / bs;
    const int tau = tt / bs, s = tt % bs;
    int u = tau <= g ? tau : tau + 3 - g;
    if (u >= nb) u += g + 1;
    return (g + 4 * (u >> 2) - (u & 3)) * bs + s;
}

template <int N> __device__ __forceinline__ bool valid(const u32x4 (&r)[N]) {
    unsigned m = ~0u;
#pragma unroll
    for (int k = 0; k < N; ++k) { m = min(m, min(r[k].x, r[k].y)); m = min(m, min(r[k].z, r[k].w)); }
    return __all(m != 0u);
}
__device__ __forceinline__ void add_bf16x16(float (&v)[16], const u32x4& lo, const u32x4& hi) {
    const unsigned wd[8] = {lo.x, lo.y, lo.z, lo.w, hi.x, hi.y, hi.z, hi.w};
#pragma unroll
    for (int i = 0; i < 8; ++i) {
        v[2 * i] += __builtin_bit_cast(float, wd[i] << 16);
        v[2 * i + 1] += __builtin_bit_cast(float, wd[i] & 0xffff0000u);
    }
}

__global__ __launch_bounds__(512, 2) void fused_kernel(Params p) {
    extern __shared__ __attribute__((aligned(16))) unsigned char shm[];
    LAS unsigned char* lds = (LAS unsigned char*)shm;
    const int tid = threadIdx.x;
    const int wid = __builtin_amdgcn_readfirstlane(tid >> 6);
    const int lane = tid & 63;
    const int b = blockIdx.x;
    const int kq = b & 3, hh = (b >> 2) & 1, idx = b >> 3;
    const int mi = idx & 3, nl = idx >> 2;
    const int tile = hh * 8 + nl;
    const int dst = tile >> 1, sub = tile & 1;
    const int T = hh * 32 + idx;
    const int m0 = mi * BM;
    const int w = wid & 3;
    const int l31 = lane & 31, h = lane >> 5;
    const unsigned ldsb = (unsigned)(size_t)lds;
    const unsigned l16 = lane * 16;
    char* myws = p.ws[p.rank];
    unsigned call = 0;
    bool bad = false;

    auto inbox = [&](int ph, int q, int j) __attribute__((always_inline)) -> unsigned {
        return (unsigned)(WS_INBOX + (size_t)(ph * 64 + T) * INBOX_TILE + (size_t)((q * 3 + (j - 1)) * 4 + w) * 2048);
    };
    auto give_up = [&](unsigned long long t0) __attribute__((always_inline)) -> bool {
        gu32* tmo = (gu32*)(myws + WS_TMO);
        if (__hip_atomic_load(tmo, __ATOMIC_RELAXED, __HIP_MEMORY_SCOPE_AGENT) == call
            || __builtin_amdgcn_s_memrealtime() - t0 > DEADLINE_TICKS) {
            __hip_atomic_store(tmo, call, __ATOMIC_RELAXED, __HIP_MEMORY_SCOPE_AGENT);
            return true;
        }
        return false;
    };
    auto finish_a = [&](int stg_off, const float (&v)[16]) __attribute__((always_inline)) {
        LAS unsigned char* stg = lds + stg_off + l31 * 256 + h * 8;
#pragma unroll
        for (int e4 = 0; e4 < 4; ++e4) {
            u32x2 o; o.x = cvt_pk(silu(v[4 * e4]), silu(v[4 * e4 + 1])); o.y = cvt_pk(silu(v[4 * e4 + 2]), silu(v[4 * e4 + 3]));
            if (bad) { o.x = 0x7fc07fc0u; o.y = 0x7fc07fc0u; }
            *(LAS u32x2*)(stg + (((w * 4 + e4) ^ (l31 & 15)) << 4)) = o;
        }
        wait_lds();
    };
    auto finish_b = [&](int ph, int stg_off) __attribute__((always_inline)) {
        const __amdgpu_buffer_rsrc_t rs_out = make_rsrc(p.out[dst], (unsigned)(ND * NCOL * 2));
        const unsigned obase = (unsigned)(((p.rank * MLOC + m0 + kq * 32) * NCOL + ph * 256 + sub * 128) * 2);
#pragma unroll
        for (int i = 0; i < 2; ++i) {
            const int r = (w * 2 + i) * 4 + (lane >> 4), c = lane & 15;
            const u32x4 v = *(LAS u32x4*)(lds + stg_off + r * 256 + ((c ^ (r & 15)) << 4));
            __builtin_amdgcn_raw_buffer_store_b128(v, rs_out, r * (NCOL * 2) + c * 16, obase, AUX_SYS);
        }
    };
    const __amdgpu_buffer_rsrc_t rs_ws = make_rsrc(myws, (unsigned)WS_END);
    auto zero_slots = [&](int ph) __attribute__((always_inline)) {
#pragma unroll
        for (int k = 0; k < 6; ++k)
            __builtin_amdgcn_raw_buffer_store_b128((u32x4){0u, 0u, 0u, 0u}, rs_ws, l16 + (k & 1) * 1024, inbox(ph, kq, 1 + (k >> 1)), AUX_SYS);
    };
    const gu32* cnt_ptr = (const gu32*)(myws + WS_CNT + (size_t)b * 128);

    if (__builtin_expect(wid >= 4, 1)) {
        unsigned callv;
        asm volatile("global_load_dword %0, %1, off sc1" : "=v"(callv) : "v"(cnt_ptr) : "memory");
        unsigned offA[4], offB[4];
#pragma unroll
        for (int i = 0; i < 4; ++i) {
            const int row = (w * 4 + i) * 8 + (lane >> 3);
            const int ch = (lane & 7) ^ ((row >> 1) & 7);
            offA[i] = (unsigned)(row * (KD * 2) + ch * 16);
            const int k = (w * 4 + i) * 4 + (lane >> 4);
            const int cb = (lane & 15) ^ ((k & 3) << 2);
            offB[i] = (unsigned)(k * (ND * 2) + cb * 16);
        }
        const char* gA = p.x + (size_t)m0 * (KD * 2) + (size_t)kq * (SPP * BK * 2);
        const char* gB = p.w + (size_t)kq * ((size_t)SPP * BK * ND * 2) + (size_t)(dst * NCOL + sub * BN) * 2;
        const int g = (mi + nl) & 3;
        struct Src { const char* a; const char* b; };
        auto source = [&](int t) __attribute__((always_inline)) -> Src {
            const int kt = korder(t & (SPP - 1), g);
            return Src{gA + kt * (BK * 2), gB + (size_t)kt * ((size_t)BK * ND * 2) + (t >> 4) * (256 * 2)};
        };
        auto issue = [&](const Src& q, int slot) __attribute__((always_inline)) {
            const unsigned la = ldsb + slot * SLOT + w * 4096;
#pragma unroll
            for (int i = 0; i < 4; ++i)
                asm volatile("s_mov_b32 m0, %2\n\ts_nop 0\n\tglobal_load_lds_dwordx4 %0, %1" :: "v"(offA[i]), "s"(q.a), "s"(la + i * 1024) : "memory");
#pragma unroll
            for (int i = 0; i < 4; ++i)
                asm volatile("s_mov_b32 m0, %2\n\ts_nop 0\n\tglobal_load_lds_dwordx4 %0, %1" :: "v"(offB[i]), "s"(q.b), "s"(la + LDS_B + i * 1024) : "memory");
        };
        auto issue_x = [&](int j) __attribute__((always_inline)) {
            const char* src = myws + inbox(0, kq, j);
#pragma unroll
            for (int i = 0; i < 2; ++i)
                asm volatile("s_mov_b32 m0, %2\n\ts_nop 0\n\tglobal_load_lds_dwordx4 %0, %1 sc0 sc1" :: "v"(l16 + i * 1024), "s"(src), "s"(ldsb + LDS_XB + (j - 1) * 8192 + w * 2048 + i * 1024) : "memory");
        };
        issue(source(0), 0);
        issue(source(1), 1);
        Src nx = source(2);
        auto step = [&](int t) __attribute__((always_inline)) {
            issue(nx, (t + 3) & (NS - 1));
            nx = source((t + 4) & (NSTEP - 1));
        };
#pragma unroll 1
        for (int t = -1; t < XT; ++t) { wait_vm<LPT>(); bar(); step(t); }
        wait_vm<LPT>(); bar();
        issue_x(1); issue_x(2); issue_x(3);
        step(XT);
        wait_vm<LPT + 6>(); bar();
        step(XT + 1);
#pragma unroll 1
        for (int t = XT + 2; t <= NSTEP - 4; ++t) { wait_vm<LPT>(); bar(); step(t); }
        wait_vm<LPT>(); bar();
        wait_vm<0>(); bar();
        bar();
        asm volatile("" : "+v"(callv));
        call = __builtin_amdgcn_readfirstlane(callv) + 1;
        bar();
        u32x4 in[6];
        {
            const unsigned long long t0 = __builtin_amdgcn_s_memrealtime();
            while (true) {
#pragma unroll
                for (int k = 0; k < 6; ++k) in[k] = __builtin_amdgcn_raw_buffer_load_b128(rs_ws, l16 + (k & 1) * 1024, inbox(1, kq, 1 + (k >> 1)), AUX_SYS);
                if (valid<6>(in)) break;
                if (give_up(t0)) { bad = true; break; }
                __builtin_amdgcn_s_sleep(TSLEEP);
            }
        }
        float v[16];
#pragma unroll
        for (int e4 = 0; e4 < 4; ++e4) {
            const f32x4 own = *(LAS f32x4*)(lds + LDS_OWN + w * 4096 + e4 * 1024 + l16);
#pragma unroll
            for (int e = 0; e < 4; ++e) v[4 * e4 + e] = own[e];
        }
        add_bf16x16(v, in[0], in[1]); add_bf16x16(v, in[2], in[3]); add_bf16x16(v, in[4], in[5]);
        finish_a(LDS_STG1, v);
        bar();
        finish_b(1, LDS_STG1);
        zero_slots(1);
        wait_vm<0>();
    } else {
        __builtin_amdgcn_s_setprio(2);
        call = __hip_atomic_load(cnt_ptr, __ATOMIC_RELAXED, __HIP_MEMORY_SCOPE_AGENT) + 1;
        unsigned xa[4][4];
#pragma unroll
        for (int j = 0; j < 4; ++j)
#pragma unroll
            for (int ks = 0; ks < 4; ++ks) xa[j][ks] = ldsb + ((j ^ kq) * 32 + l31) * 128 + (((2 * ks + h) ^ ((l31 >> 1) & 7)) << 4);
        const int blk = (lane >> 4) & 1, q4 = (lane & 15) >> 2, p4 = lane & 3;
        unsigned wa = ldsb + LDS_B + (8 * h + q4) * 256 + (((2 * w + blk) ^ (q4 << 1)) << 5) + 8 * p4;
        auto rd1 = [&](F& f, int so, int hf, int r) __attribute__((always_inline)) {
            const int s = r / 6, rr = r % 6, ks = 2 * hf + s;
            if (rr < 2) f.b[s][rr] = __builtin_amdgcn_ds_read_tr16_b64_v4i16((LAS bf16x4*)(size_t)(wa + so + ks * 4096 + rr * 1024));
            else f.a[s][rr - 2] = *(LAS bf16x8*)(size_t)(xa[rr - 2][ks] + so);
        };
        auto half = [&](bool ph1, const F& cur, F& nxt, int so, int hf) __attribute__((always_inline)) {
#pragma unroll
            for (int s = 0; s < 2; ++s) {
                const bf16x8 wv = __builtin_shufflevector(cur.b[s][0], cur.b[s][1], 0, 1, 2, 3, 4, 5, 6, 7);
#pragma unroll
                for (int j = 0; j < 4; ++j) {
#define MFMA_A(R, CL) asm volatile("v_mfma_f32_32x32x16_bf16 " R ", %0, %1, " R :: "v"(wv), "v"(cur.a[s][j]) : CL)
                    if (ph1) {
                        if (j == 0) MFMA_A("a[0:15]", ACLOB0); else if (j == 1) MFMA_A("a[16:31]", ACLOB1);
                        else if (j == 2) MFMA_A("a[32:47]", ACLOB2); else MFMA_A("a[48:63]", ACLOB3);
                    } else {
                        if (j == 0) MFMA_A("a[64:79]", ACLOB4); else if (j == 1) MFMA_A("a[80:95]", ACLOB5);
                        else if (j == 2) MFMA_A("a[96:111]", ACLOB6); else MFMA_A("a[112:127]", ACLOB7);
                    }
#undef MFMA_A
                    SCHED();
                    const int i = s * 4 + j;
                    const int r0 = i < 4 ? 2 * i : 4 + i, n = i < 4 ? 2 : 1;
                    rd1(nxt, so, hf, r0); if (n == 2) rd1(nxt, so, hf, r0 + 1);
                    SCHED();
                }
            }
        };
        auto toggle = [&]() __attribute__((always_inline)) {
#pragma unroll
            for (int j = 0; j < 4; ++j)
#pragma unroll
                for (int ks = 0; ks < 4; ++ks) xa[j][ks] ^= 2 * SLOT;
            wa ^= 2 * SLOT;
        };
        F f0, f1;
        auto pair = [&](bool ph1, auto&& actA, auto&& actB) __attribute__((always_inline)) {
            bar();
            actA(); SCHED();
            half(ph1, f0, f1, 0, 1);
            half(ph1, f1, f0, SLOT, 0);
            bar();
            actB(); SCHED();
            half(ph1, f0, f1, SLOT, 1);
            toggle();
            SCHED();
            half(ph1, f1, f0, 0, 0);
        };
        auto none = [&]() __attribute__((always_inline)) {};
        auto send_a = [&](auto pc, auto jc) __attribute__((always_inline)) {
            constexpr int ph = decltype(pc)::value, j = decltype(jc)::value, R = 64 * ph + 16 * j;
            const unsigned off = inbox(ph, kq ^ j, j);
            auto pk = [&](f32x4 x, int k) __attribute__((always_inline)) -> unsigned { const unsigned wd = cvt_pk(x[2 * k], x[2 * k + 1]); return wd | ((wd - 1u) & 0x80000000u); };
            { const f32x4 a = areg_quad<R>(), c = areg_quad<R + 4>();
              __builtin_amdgcn_raw_buffer_store_b128((u32x4){pk(a, 0), pk(a, 1), pk(c, 0), pk(c, 1)}, rs_ws, l16, off, AUX_SYS); }
            { const f32x4 a = areg_quad<R + 8>(), c = areg_quad<R + 12>();
              __builtin_amdgcn_raw_buffer_store_b128((u32x4){pk(a, 0), pk(a, 1), pk(c, 0), pk(c, 1)}, rs_ws, l16 + 1024, off, AUX_SYS); }
        };
        constexpr std::integral_constant<int, 0> I0{}; constexpr std::integral_constant<int, 1> I1{};
        constexpr std::integral_constant<int, 2> I2{}; constexpr std::integral_constant<int, 3> I3{};
        auto finish_ph1 = [&]() __attribute__((always_inline)) {
            u32x4 r[6];
#pragma unroll
            for (int k = 0; k < 6; ++k) r[k] = *(LAS u32x4*)(lds + LDS_XB + (k >> 1) * 8192 + w * 2048 + (k & 1) * 1024 + l16);
            float v[16];
            { const f32x4 q0 = areg_quad<0>(), q1 = areg_quad<4>(), q2 = areg_quad<8>(), q3 = areg_quad<12>();
#pragma unroll
              for (int e = 0; e < 4; ++e) { v[e] = q0[e]; v[4 + e] = q1[e]; v[8 + e] = q2[e]; v[12 + e] = q3[e]; } }
            if (__builtin_expect(!valid<6>(r), 0)) {
                const unsigned long long t0 = __builtin_amdgcn_s_memrealtime();
                while (true) {
#pragma unroll
                    for (int k = 0; k < 6; ++k) r[k] = __builtin_amdgcn_raw_buffer_load_b128(rs_ws, l16 + (k & 1) * 1024, inbox(0, kq, 1 + (k >> 1)), AUX_SYS);
                    if (valid<6>(r)) break;
                    if (give_up(t0)) { bad = true; break; }
                }
            }
            add_bf16x16(v, r[0], r[1]); add_bf16x16(v, r[2], r[3]); add_bf16x16(v, r[4], r[5]);
            finish_a(LDS_STG0, v);
        };

        asm volatile(
#define Z4(a, b, c, d) "v_accvgpr_write_b32 a" #a ", 0\n\tv_accvgpr_write_b32 a" #b ", 0\n\tv_accvgpr_write_b32 a" #c ", 0\n\tv_accvgpr_write_b32 a" #d ", 0\n\t"
            Z4(0,1,2,3) Z4(4,5,6,7) Z4(8,9,10,11) Z4(12,13,14,15) Z4(16,17,18,19) Z4(20,21,22,23) Z4(24,25,26,27) Z4(28,29,30,31) Z4(32,33,34,35) Z4(36,37,38,39) Z4(40,41,42,43) Z4(44,45,46,47) Z4(48,49,50,51) Z4(52,53,54,55) Z4(56,57,58,59) Z4(60,61,62,63) Z4(64,65,66,67) Z4(68,69,70,71) Z4(72,73,74,75) Z4(76,77,78,79) Z4(80,81,82,83) Z4(84,85,86,87) Z4(88,89,90,91) Z4(92,93,94,95) Z4(96,97,98,99) Z4(100,101,102,103) Z4(104,105,106,107) Z4(108,109,110,111) Z4(112,113,114,115) Z4(116,117,118,119) Z4(120,121,122,123) Z4(124,125,126,127)
#undef Z4
            "s_nop 1" ::: ACLOB0, ACLOB1, ACLOB2, ACLOB3, ACLOB4, ACLOB5, ACLOB6, ACLOB7);
        bar();
#pragma unroll
        for (int r = 0; r < 12; ++r) rd1(f0, 0, 0, r);
#pragma unroll 1
        for (int u = 0; u < SPP / 2; ++u) {
            pair(true, none, none);
        }
        constexpr int UF = (XT + 3 - 16) / 2;
        const int cls = NCLS == 8 ? mi + 4 * (w & 1) : mi;
        const int sstep = XT + 4 + cls - 16;
        const int stA = (sstep & 1) ? -1 : sstep >> 1, stB = (sstep & 1) ? sstep >> 1 : -1;
#pragma unroll 1
        for (int u = 0; u < SPP / 2; ++u) {
            pair(false,
                 [&]() __attribute__((always_inline)) {
                     if (__builtin_expect(u == 0 || u == UF || u == stA, 0)) {
                         if (u == 0) {
                             SCHED(); asm volatile("s_nop 7\n\ts_nop 7" ::: "memory"); SCHED();
                             send_a(I0, I1); send_a(I0, I2); send_a(I0, I3);
                         } else if (u == UF) finish_ph1();
                         else finish_b(0, LDS_STG0);
                     }
                 },
                 [&]() __attribute__((always_inline)) {
                     if (__builtin_expect(u == stB, 0)) finish_b(0, LDS_STG0);
                 });
        }
        SCHED(); asm volatile("s_nop 7\n\ts_nop 7" ::: "memory"); SCHED();
        send_a(I1, I1); send_a(I1, I2); send_a(I1, I3);
        {
            LAS unsigned char* o = lds + LDS_OWN + w * 4096 + l16;
            *(LAS f32x4*)(o) = areg_quad<64>(); *(LAS f32x4*)(o + 1024) = areg_quad<68>(); *(LAS f32x4*)(o + 2048) = areg_quad<72>(); *(LAS f32x4*)(o + 3072) = areg_quad<76>();
        }
        wait_lds();
        bar();
        bar();
        zero_slots(0);
        wait_vm<0>();
    }
    bar();
    if (wid == 0) {
        const int src = p.rank;
        gu32* cnt = (gu32*)(myws + WS_CNT + (size_t)b * 128);
        if (lane == 0) {
            gu32* flag = (gu32*)(p.ws[dst] + WS_FLAGS + (size_t)(src * 32 + kq * 8 + mi * 2 + sub) * 128);
            __hip_atomic_store(flag, call, __ATOMIC_RELAXED, __HIP_MEMORY_SCOPE_SYSTEM);
            __hip_atomic_store(cnt, call, __ATOMIC_RELAXED, __HIP_MEMORY_SCOPE_AGENT);
            gu32* mine = (gu32*)(myws + WS_FLAGS + (size_t)b * 128);
            const unsigned long long t0 = __builtin_amdgcn_s_memrealtime();
            while ((int)(__hip_atomic_load(mine, __ATOMIC_RELAXED, __HIP_MEMORY_SCOPE_SYSTEM) - call) < 0) {
                if (give_up(t0)) {
                    gu16* o = (gu16*)(p.out[p.rank] + (size_t)p.rank * MLOC * NCOL * 2);
                    __hip_atomic_store(o, (unsigned short)0x7fc0, __ATOMIC_RELAXED, __HIP_MEMORY_SCOPE_SYSTEM);
                    break;
                }
                __builtin_amdgcn_s_sleep(1);
            }
        }
    }
}

}

extern "C" void kernel_launch(int rank, int nranks, ncclComm_t comm, hipStream_t stream,
                              void* const* d_in, const int* in_sizes, int n_in,
                              void* d_out, int out_size, const int* in_dtypes, int out_dtype,
                              void* const* d_out_all, void* const* d_ws_all, size_t ws_bytes) {
    (void)comm; (void)in_sizes; (void)n_in; (void)d_out; (void)out_size; (void)in_dtypes; (void)out_dtype;
    if (nranks != NR || rank < 0 || rank >= NR || ws_bytes < WS_END) { fprintf(stderr, "unsupported configuration\n"); return; }
    static bool attr = false;
    if (!attr) {
        HIP_CHECK(hipFuncSetAttribute((const void*)fused_kernel, hipFuncAttributeMaxDynamicSharedMemorySize, LDS_BYTES));
        attr = true;
    }
    Params p;
    p.x = (const char*)d_in[0];
    p.w = (const char*)d_in[1];
    for (int i = 0; i < NR; ++i) { p.out[i] = (char*)d_out_all[i]; p.ws[i] = (char*)d_ws_all[i]; }
    p.rank = rank;
    p.pad = 0;
    hipLaunchKernelGGL(fused_kernel, dim3(NWG), dim3(512), LDS_BYTES, stream, p);
    HIP_CHECK(hipGetLastError());
}
```

```cpp
#include <hip/hip_runtime.h>
#include <rccl/rccl.h>
#include <cstdio>
#include <type_traits>

#define HIP_CHECK(expr) do { hipError_t _e = (expr); if (_e != hipSuccess) { \
    fprintf(stderr, "HIP error %s at %s:%d\n", hipGetErrorString(_e), __FILE__, __LINE__); \
    return; } } while (0)

namespace {

#define LAS __attribute__((address_space(3)))
#define GAS __attribute__((address_space(1)))
typedef short bf16x8 __attribute__((ext_vector_type(8)));
typedef short bf16x4 __attribute__((ext_vector_type(4)));
typedef float f32x4 __attribute__((ext_vector_type(4)));
typedef float f32x16 __attribute__((ext_vector_type(16)));
typedef unsigned u32x4 __attribute__((ext_vector_type(4)));
typedef unsigned u32x2 __attribute__((ext_vector_type(2)));
typedef GAS unsigned gu32;
typedef GAS unsigned short gu16;

constexpr int NR = 8;
constexpr int MLOC = 512;
constexpr int KD = 4096, ND = 4096;
constexpr int NCOL = ND / NR;
constexpr int BM = 128, BN = 128, BK = 64;
constexpr int SPP = 16;
constexpr int NSTEP = 2 * SPP;
constexpr int SLOT = 16384;
constexpr int NS = 4;
constexpr int LDS_B = NS * SLOT;
constexpr int LDS_XB = 2 * NS * SLOT;
constexpr int LDS_STG0 = LDS_XB + 3 * 8192;
constexpr int LDS_OWN = 0;
constexpr int LDS_STG1 = 16384;
constexpr int LDS_BYTES = LDS_STG0 + 8192;
constexpr int NWG = 256;
constexpr int LPT = 8;
#define XT 19
#define NCLS 8
#define TSLEEP 4
#define FBM 3
#define LEAD_BS 1

constexpr size_t WS_FLAGS = 0;
constexpr size_t WS_CNT = WS_FLAGS + (size_t)NWG * 128;
constexpr size_t WS_TMO = WS_CNT + (size_t)NWG * 128;
constexpr size_t WS_STAMP = WS_TMO + 128;
constexpr size_t WS_INBOX = 1u << 20;
constexpr size_t INBOX_TILE = (size_t)4 * 3 * 4 * 2 * 1024;
constexpr size_t WS_INBOX_BYTES = 2 * 64 * INBOX_TILE;
constexpr size_t WS_END = WS_INBOX + WS_INBOX_BYTES;
static_assert(WS_STAMP + (size_t)NWG * 128 <= WS_INBOX, "layout");
constexpr unsigned long long DEADLINE_TICKS = 20000000ull;

struct Params {
    const char* x;
    const char* w;
    char* out[NR];
    char* ws[NR];
    int rank;
    int pad;
};

template <int N> __device__ __forceinline__ void wait_vm() { asm volatile("s_waitcnt vmcnt(%0)" :: "n"(N) : "memory"); }
__device__ __forceinline__ void wait_lds() { asm volatile("s_waitcnt lgkmcnt(0)" ::: "memory"); }
__device__ __forceinline__ void bar() { asm volatile("" ::: "memory"); __builtin_amdgcn_s_barrier(); asm volatile("" ::: "memory"); }
#define SCHED() __builtin_amdgcn_sched_barrier(0)
#define ACLOB0 "a0","a1","a2","a3","a4","a5","a6","a7","a8","a9","a10","a11","a12","a13","a14","a15"
#define ACLOB1 "a16","a17","a18","a19","a20","a21","a22","a23","a24","a25","a26","a27","a28","a29","a30","a31"
#define ACLOB2 "a32","a33","a34","a35","a36","a37","a38","a39","a40","a41","a42","a43","a44","a45","a46","a47"
#define ACLOB3 "a48","a49","a50","a51","a52","a53","a54","a55","a56","a57","a58","a59","a60","a61","a62","a63"
#define ACLOB4 "a64","a65","a66","a67","a68","a69","a70","a71","a72","a73","a74","a75","a76","a77","a78","a79"
#define ACLOB5 "a80","a81","a82","a83","a84","a85","a86","a87","a88","a89","a90","a91","a92","a93","a94","a95"
#define ACLOB6 "a96","a97","a98","a99","a100","a101","a102","a103","a104","a105","a106","a107","a108","a109","a110","a111"
#define ACLOB7 "a112","a113","a114","a115","a116","a117","a118","a119","a120","a121","a122","a123","a124","a125","a126","a127"
template <int R> __device__ __forceinline__ float areg_read() { float v; asm volatile("v_accvgpr_read_b32 %0, a%c1" : "=v"(v) : "i"(R)); return v; }
template <int R> __device__ __forceinline__ void areg_write(float v) { asm volatile("v_accvgpr_write_b32 a%c0, %1" :: "i"(R), "v"(v)); }
template <int R> __device__ __forceinline__ void areg_read16(float (&v)[16]) {
    asm volatile("s_nop 1\n\tv_accvgpr_read_b32 %0, a%c16\n\tv_accvgpr_read_b32 %1, a%c17\n\tv_accvgpr_read_b32 %2, a%c18\n\tv_accvgpr_read_b32 %3, a%c19\n\t"
                 "v_accvgpr_read_b32 %4, a%c20\n\tv_accvgpr_read_b32 %5, a%c21\n\tv_accvgpr_read_b32 %6, a%c22\n\tv_accvgpr_read_b32 %7, a%c23\n\t"
                 "v_accvgpr_read_b32 %8, a%c24\n\tv_accvgpr_read_b32 %9, a%c25\n\tv_accvgpr_read_b32 %10, a%c26\n\tv_accvgpr_read_b32 %11, a%c27\n\t"
                 "v_accvgpr_read_b32 %12, a%c28\n\tv_accvgpr_read_b32 %13, a%c29\n\tv_accvgpr_read_b32 %14, a%c30\n\tv_accvgpr_read_b32 %15, a%c31"
                 : "=v"(v[0]), "=v"(v[1]), "=v"(v[2]), "=v"(v[3]), "=v"(v[4]), "=v"(v[5]), "=v"(v[6]), "=v"(v[7]),
                   "=v"(v[8]), "=v"(v[9]), "=v"(v[10]), "=v"(v[11]), "=v"(v[12]), "=v"(v[13]), "=v"(v[14]), "=v"(v[15])
                 : "i"(R), "i"(R + 1), "i"(R + 2), "i"(R + 3), "i"(R + 4), "i"(R + 5), "i"(R + 6), "i"(R + 7),
                   "i"(R + 8), "i"(R + 9), "i"(R + 10), "i"(R + 11), "i"(R + 12), "i"(R + 13), "i"(R + 14), "i"(R + 15));
}
template <int B> __device__ __forceinline__ f32x4 areg_quad() { return (f32x4){areg_read<B>(), areg_read<B + 1>(), areg_read<B + 2>(), areg_read<B + 3>()}; }

__device__ __forceinline__ float silu(float y) {
    const float c = __builtin_amdgcn_fmed3f(y, -60.0f, 60.0f);
    const float e = __builtin_amdgcn_exp2f(c * -1.44269504088896340736f);
    return y * __builtin_amdgcn_rcpf(1.0f + e);
}
__device__ __forceinline__ unsigned cvt_pk(float lo, float hi) {
    unsigned r; asm("v_cvt_pk_bf16_f32 %0, %1, %2" : "=v"(r) : "v"(lo), "v"(hi)); return r;
}
__device__ __forceinline__ __amdgpu_buffer_rsrc_t make_rsrc(const char* base, unsigned bytes) {
    return __builtin_amdgcn_make_buffer_rsrc((void*)base, 0, bytes, 0x00020000);
}
__device__ __forceinline__ f32x4 quad(const f32x16& v, int e4) { return (f32x4){v[4 * e4], v[4 * e4 + 1], v[4 * e4 + 2], v[4 * e4 + 3]}; }
constexpr int AUX_SYS = 17;
#define AUX_XS 17
#define XL_BITS "sc0 sc1"
#define AUX_XL 17

struct F { bf16x8 a[2][4]; bf16x4 b[2][2]; };

__device__ __forceinline__ int korder(int tt, int g) {
    constexpr int bs = LEAD_BS, nb = SPP / bs;
    const int tau = tt / bs, s = tt % bs;
    int u = tau <= g ? tau : tau + 3 - g;
    if (u >= nb) u += g + 1;
    return (g + 4 * (u >> 2) - (u & 3)) * bs + s;
}

template <int N> __device__ __forceinline__ bool valid(const u32x4 (&r)[N]) {
    unsigned m = ~0u;
#pragma unroll
    for (int k = 0; k < N; ++k) { m = min(m, min(r[k].x, r[k].y)); m = min(m, min(r[k].z, r[k].w)); }
    return __all(m != 0u);
}
__device__ __forceinline__ void add_bf16x16(float (&v)[16], const u32x4& lo, const u32x4& hi) {
    const unsigned wd[8] = {lo.x, lo.y, lo.z, lo.w, hi.x, hi.y, hi.z, hi.w};
#pragma unroll
    for (int i = 0; i < 8; ++i) {
        v[2 * i] += __builtin_bit_cast(float, wd[i] << 16);
        v[2 * i + 1] += __builtin_bit_cast(float, wd[i] & 0xffff0000u);
    }
}

__global__ __launch_bounds__(512, 2) void fused_kernel(Params p) {
    extern __shared__ __attribute__((aligned(16))) unsigned char shm[];
    LAS unsigned char* lds = (LAS unsigned char*)shm;
    const int tid = threadIdx.x;
    const int wid = __builtin_amdgcn_readfirstlane(tid >> 6);
    const int lane = tid & 63;
    const int b = blockIdx.x;
    const int kq = b & 3, hh = (b >> 2) & 1, idx = b >> 3;
    const int mi = idx & 3, nl = idx >> 2;
    const int tile = hh * 8 + nl;
    const int dst = tile >> 1, sub = tile & 1;
    const int T = hh * 32 + idx;
    const int m0 = mi * BM;
    const int w = wid & 3;
    const int l31 = lane & 31, h = lane >> 5;
    const unsigned ldsb = (unsigned)(size_t)lds;
    const unsigned l16 = lane * 16;
    char* myws = p.ws[p.rank];
    unsigned call = 0;
    bool bad = false;

    auto inbox = [&](int ph, int q, int j) __attribute__((always_inline)) -> unsigned {
        return (unsigned)(WS_INBOX + (size_t)(ph * 64 + T) * INBOX_TILE + (size_t)((q * 3 + (j - 1)) * 4 + w) * 2048);
    };
    auto give_up = [&](unsigned long long t0) __attribute__((always_inline)) -> bool {
        gu32* tmo = (gu32*)(myws + WS_TMO);
        if (__hip_atomic_load(tmo, __ATOMIC_RELAXED, __HIP_MEMORY_SCOPE_AGENT) == call
            || __builtin_amdgcn_s_memrealtime() - t0 > DEADLINE_TICKS) {
            __hip_atomic_store(tmo, call, __ATOMIC_RELAXED, __HIP_MEMORY_SCOPE_AGENT);
            return true;
        }
        return false;
    };
    auto finish_a = [&](int stg_off, const float (&v)[16]) __attribute__((always_inline)) {
        LAS unsigned char* stg = lds + stg_off + l31 * 256 + h * 8;
        unsigned o[8];
#pragma unroll
        for (int i = 0; i < 8; ++i) o[i] = cvt_pk(silu(v[2 * i]), silu(v[2 * i + 1]));
        if (__builtin_expect(bad, 0)) {
#pragma unroll
            for (int i = 0; i < 8; ++i) o[i] = 0x7fc07fc0u;
        }
#pragma unroll
        for (int e4 = 0; e4 < 4; ++e4) *(LAS u32x2*)(stg + (((w * 4 + e4) ^ (l31 & 15)) << 4)) = (u32x2){o[2 * e4], o[2 * e4 + 1]};
        wait_lds();
    };
    const __amdgpu_buffer_rsrc_t rs_out = make_rsrc(p.out[dst], (unsigned)(ND * NCOL * 2));
    auto finish_b = [&](int ph, int stg_off) __attribute__((always_inline)) {
        const unsigned obase = (unsigned)(((p.rank * MLOC + m0 + kq * 32) * NCOL + ph * 256 + sub * 128) * 2);
#pragma unroll
        for (int i = 0; i < 2; ++i) {
            const int r = (w * 2 + i) * 4 + (lane >> 4), c = lane & 15;
            const u32x4 v = *(LAS u32x4*)(lds + stg_off + r * 256 + ((c ^ (r & 15)) << 4));
            __builtin_amdgcn_raw_buffer_store_b128(v, rs_out, r * (NCOL * 2) + c * 16, obase, AUX_SYS);
        }
    };
    const __amdgpu_buffer_rsrc_t rs_ws = make_rsrc(myws, (unsigned)WS_END);
    auto zero_slots = [&](int ph) __attribute__((always_inline)) {
#pragma unroll
        for (int k = 0; k < 6; ++k)
            __builtin_amdgcn_raw_buffer_store_b128((u32x4){0u, 0u, 0u, 0u}, rs_ws, l16 + (k & 1) * 1024, inbox(ph, kq, 1 + (k >> 1)), AUX_XS);
    };
    const gu32* cnt_ptr = (const gu32*)(myws + WS_CNT + (size_t)b * 128);

    if (__builtin_expect(wid >= 4, 1)) {
        unsigned callv;
        asm volatile("global_load_dword %0, %1, off sc1" : "=v"(callv) : "v"(cnt_ptr) : "memory");
        unsigned offA[4], offB[4];
#pragma unroll
        for (int i = 0; i < 4; ++i) {
            const int row = (w * 4 + i) * 8 + (lane >> 3);
            const int ch = (lane & 7) ^ ((row >> 1) & 7);
            offA[i] = (unsigned)(row * (KD * 2) + ch * 16);
            const int k = (w * 4 + i) * 4 + (lane >> 4);
            const int cb = (lane & 15) ^ ((k & 3) << 2);
            offB[i] = (unsigned)(k * (ND * 2) + cb * 16);
        }
        const char* gA = p.x + (size_t)m0 * (KD * 2) + (size_t)kq * (SPP * BK * 2);
        const char* gB = p.w + (size_t)kq * ((size_t)SPP * BK * ND * 2) + (size_t)(dst * NCOL + sub * BN) * 2;
        const int g = (mi + nl) & 3;
        struct Src { const char* a; const char* b; };
        auto source = [&](int t) __attribute__((always_inline)) -> Src {
            const int kt = korder(t & (SPP - 1), g);
            return Src{gA + kt * (BK * 2), gB + (size_t)kt * ((size_t)BK * ND * 2) + (t >> 4) * (256 * 2)};
        };
        auto issue = [&](const Src& q, int slot) __attribute__((always_inline)) {
            const unsigned la = ldsb + slot * SLOT + w * 4096;
#pragma unroll
            for (int i = 0; i < 4; ++i)
                asm volatile("s_mov_b32 m0, %2\n\ts_nop 0\n\tglobal_load_lds_dwordx4 %0, %1" :: "v"(offA[i]), "s"(q.a), "s"(la + i * 1024) : "memory");
#pragma unroll
            for (int i = 0; i < 4; ++i)
                asm volatile("s_mov_b32 m0, %2\n\ts_nop 0\n\tglobal_load_lds_dwordx4 %0, %1" :: "v"(offB[i]), "s"(q.b), "s"(la + LDS_B + i * 1024) : "memory");
        };
        auto issue_x = [&](int j) __attribute__((always_inline)) {
            const char* src = myws + inbox(0, kq, j);
#pragma unroll
            for (int i = 0; i < 2; ++i)
                asm volatile("s_mov_b32 m0, %2\n\ts_nop 0\n\tglobal_load_lds_dwordx4 %0, %1 " XL_BITS :: "v"(l16 + i * 1024), "s"(src), "s"(ldsb + LDS_XB + (j - 1) * 8192 + w * 2048 + i * 1024) : "memory");
        };
        issue(source(0), 0);
        issue(source(1), 1);
        Src nx = source(2);
        auto step = [&](int t) __attribute__((always_inline)) {
            issue(nx, (t + 3) & (NS - 1));
            nx = source((t + 4) & (NSTEP - 1));
        };
#pragma unroll 1
        for (int t = -1; t < XT; ++t) { wait_vm<LPT>(); bar(); step(t); }
        wait_vm<LPT>(); bar();
        issue_x(1); issue_x(2); issue_x(3);
        step(XT);
        wait_vm<LPT + 6>(); bar();
        step(XT + 1);
#pragma unroll 1
        for (int t = XT + 2; t <= NSTEP - 4; ++t) { wait_vm<LPT>(); bar(); step(t); }
        wait_vm<LPT>(); bar();
        wait_vm<0>(); bar();
        bar();
        bar();
        asm volatile("" : "+v"(callv));
        call = __builtin_amdgcn_readfirstlane(callv) + 1;
        u32x4 in[6];
        {
            const unsigned long long t0 = __builtin_amdgcn_s_memrealtime();
            while (true) {
#pragma unroll
                for (int k = 0; k < 6; ++k) in[k] = __builtin_amdgcn_raw_buffer_load_b128(rs_ws, l16 + (k & 1) * 1024, inbox(1, kq, 1 + (k >> 1)), AUX_XL);
                if (valid<6>(in)) break;
                if (give_up(t0)) { bad = true; break; }
                __builtin_amdgcn_s_sleep(TSLEEP);
            }
        }
        float v[16];
#pragma unroll
        for (int e4 = 0; e4 < 4; ++e4) {
            const f32x4 own = *(LAS f32x4*)(lds + LDS_OWN + w * 4096 + e4 * 1024 + l16);
#pragma unroll
            for (int e = 0; e < 4; ++e) v[4 * e4 + e] = own[e];
        }
        add_bf16x16(v, in[0], in[1]); add_bf16x16(v, in[2], in[3]); add_bf16x16(v, in[4], in[5]);
        finish_a(LDS_STG1, v);
        bar();
        finish_b(1, LDS_STG1);
        zero_slots(1);
        wait_vm<0>();
    } else {
        __builtin_amdgcn_s_setprio(2);
        call = __hip_atomic_load(cnt_ptr, __ATOMIC_RELAXED, __HIP_MEMORY_SCOPE_AGENT) + 1;
        unsigned xa[4][4];
#pragma unroll
        for (int j = 0; j < 4; ++j)
#pragma unroll
            for (int ks = 0; ks < 4; ++ks) xa[j][ks] = ldsb + ((j ^ kq) * 32 + l31) * 128 + (((2 * ks + h) ^ ((l31 >> 1) & 7)) << 4);
        const int blk = (lane >> 4) & 1, q4 = (lane & 15) >> 2, p4 = lane & 3;
        unsigned wa = ldsb + LDS_B + (8 * h + q4) * 256 + (((2 * w + blk) ^ (q4 << 1)) << 5) + 8 * p4;
        auto rd1 = [&](F& f, int so, int hf, int r) __attribute__((always_inline)) {
            const int s = r / 6, rr = r % 6, ks = 2 * hf + s;
            if (rr < 2) f.b[s][rr] = __builtin_amdgcn_ds_read_tr16_b64_v4i16((LAS bf16x4*)(size_t)(wa + so + ks * 4096 + rr * 1024));
            else f.a[s][rr - 2] = *(LAS bf16x8*)(size_t)(xa[rr - 2][ks] + so);
        };
        auto half = [&](bool ph1, const F& cur, F& nxt, int so, int hf) __attribute__((always_inline)) {
#pragma unroll
            for (int s = 0; s < 2; ++s) {
                const bf16x8 wv = __builtin_shufflevector(cur.b[s][0], cur.b[s][1], 0, 1, 2, 3, 4, 5, 6, 7);
#pragma unroll
                for (int j = 0; j < 4; ++j) {
#define MFMA_A(R, CL) asm volatile("v_mfma_f32_32x32x16_bf16 " R ", %0, %1, " R :: "v"(wv), "v"(cur.a[s][j]) : CL)
                    if (ph1) {
                        if (j == 0) MFMA_A("a[0:15]", ACLOB0); else if (j == 1) MFMA_A("a[16:31]", ACLOB1);
                        else if (j == 2) MFMA_A("a[32:47]", ACLOB2); else MFMA_A("a[48:63]", ACLOB3);
                    } else {
                        if (j == 0) MFMA_A("a[64:79]", ACLOB4); else if (j == 1) MFMA_A("a[80:95]", ACLOB5);
                        else if (j == 2) MFMA_A("a[96:111]", ACLOB6); else MFMA_A("a[112:127]", ACLOB7);
                    }
#undef MFMA_A
                    SCHED();
                    const int i = s * 4 + j;
                    const int r0 = i < 4 ? 2 * i : 4 + i, n = i < 4 ? 2 : 1;
                    rd1(nxt, so, hf, r0); if (n == 2) rd1(nxt, so, hf, r0 + 1);
                    SCHED();
                }
            }
        };
        auto toggle = [&]() __attribute__((always_inline)) {
#pragma unroll
            for (int j = 0; j < 4; ++j)
#pragma unroll
                for (int ks = 0; ks < 4; ++ks) xa[j][ks] ^= 2 * SLOT;
            wa ^= 2 * SLOT;
        };
        F f0, f1;
        auto pair = [&](bool ph1, auto&& actA, auto&& actB) __attribute__((always_inline)) {
            bar();
            actA(); SCHED();
            half(ph1, f0, f1, 0, 1);
            half(ph1, f1, f0, SLOT, 0);
            bar();
            actB(); SCHED();
            half(ph1, f0, f1, SLOT, 1);
            toggle();
            SCHED();
            half(ph1, f1, f0, 0, 0);
        };
        auto none = [&]() __attribute__((always_inline)) {};
        auto send_a = [&](auto pc, auto jc) __attribute__((always_inline)) {
            constexpr int ph = decltype(pc)::value, j = decltype(jc)::value, R = 64 * ph + 16 * j;
            const unsigned off = inbox(ph, kq ^ j, j);
            float v[16];
            areg_read16<R>(v);
            unsigned wd[8];
#pragma unroll
            for (int i = 0; i < 8; ++i) wd[i] = cvt_pk(v[2 * i], v[2 * i + 1]);
            const unsigned m = min(min(min(wd[0], wd[1]), min(wd[2], wd[3])), min(min(wd[4], wd[5]), min(wd[6], wd[7])));
            if (__builtin_expect(__any(m == 0u), 0)) {
#pragma unroll
                for (int i = 0; i < 8; ++i) wd[i] |= (wd[i] - 1u) & 0x80000000u;
            }
            __builtin_amdgcn_raw_buffer_store_b128((u32x4){wd[0], wd[1], wd[2], wd[3]}, rs_ws, l16, off, AUX_XS);
            __builtin_amdgcn_raw_buffer_store_b128((u32x4){wd[4], wd[5], wd[6], wd[7]}, rs_ws, l16 + 1024, off, AUX_XS);
        };
        constexpr std::integral_constant<int, 0> I0{}; constexpr std::integral_constant<int, 1> I1{};
        constexpr std::integral_constant<int, 2> I2{}; constexpr std::integral_constant<int, 3> I3{};
        auto finish_ph1 = [&]() __attribute__((always_inline)) {
            u32x4 r[6];
#pragma unroll
            for (int k = 0; k < 6; ++k) r[k] = *(LAS u32x4*)(lds + LDS_XB + (k >> 1) * 8192 + w * 2048 + (k & 1) * 1024 + l16);
            float v[16];
            areg_read16<0>(v);
            if (__builtin_expect(!valid<6>(r), 0)) {
                const unsigned long long t0 = __builtin_amdgcn_s_memrealtime();
                while (true) {
#pragma unroll
                    for (int k = 0; k < 6; ++k) r[k] = __builtin_amdgcn_raw_buffer_load_b128(rs_ws, l16 + (k & 1) * 1024, inbox(0, kq, 1 + (k >> 1)), AUX_XL);
                    if (valid<6>(r)) break;
                    if (give_up(t0)) { bad = true; break; }
                }
            }
            add_bf16x16(v, r[0], r[1]); add_bf16x16(v, r[2], r[3]); add_bf16x16(v, r[4], r[5]);
            finish_a(LDS_STG0, v);
        };

        asm volatile(
#define Z4(a, b, c, d) "v_accvgpr_write_b32 a" #a ", 0\n\tv_accvgpr_write_b32 a" #b ", 0\n\tv_accvgpr_write_b32 a" #c ", 0\n\tv_accvgpr_write_b32 a" #d ", 0\n\t"
            Z4(0,1,2,3) Z4(4,5,6,7) Z4(8,9,10,11) Z4(12,13,14,15) Z4(16,17,18,19) Z4(20,21,22,23) Z4(24,25,26,27) Z4(28,29,30,31) Z4(32,33,34,35) Z4(36,37,38,39) Z4(40,41,42,43) Z4(44,45,46,47) Z4(48,49,50,51) Z4(52,53,54,55) Z4(56,57,58,59) Z4(60,61,62,63) Z4(64,65,66,67) Z4(68,69,70,71) Z4(72,73,74,75) Z4(76,77,78,79) Z4(80,81,82,83) Z4(84,85,86,87) Z4(88,89,90,91) Z4(92,93,94,95) Z4(96,97,98,99) Z4(100,101,102,103) Z4(104,105,106,107) Z4(108,109,110,111) Z4(112,113,114,115) Z4(116,117,118,119) Z4(120,121,122,123) Z4(124,125,126,127)
#undef Z4
            "s_nop 1" ::: ACLOB0, ACLOB1, ACLOB2, ACLOB3, ACLOB4, ACLOB5, ACLOB6, ACLOB7);
        bar();
#pragma unroll
        for (int r = 0; r < 12; ++r) rd1(f0, 0, 0, r);
#pragma unroll 1
        for (int u = 0; u < SPP / 2; ++u) {
            pair(true, none, none);
        }
        constexpr int UF = (XT + 3 - 16) / 2;
        const int cls = NCLS == 8 ? mi + 4 * (w & 1) : mi;
        const int sstep = XT + 4 + cls - 16;
        unsigned maskA = 1u | (1u << UF) | ((sstep & 1) ? 0u : 1u << (sstep >> 1)), maskB = (sstep & 1) ? 1u << (sstep >> 1) : 0u;
        asm volatile("" : "+s"(maskA), "+s"(maskB));
#pragma unroll 1
        for (int u = 0; u < SPP / 2; ++u) {
            pair(false,
                 [&]() __attribute__((always_inline)) {
                     if (__builtin_expect((maskA >> u) & 1u, 0)) {
                         if (u == 0) {
                             SCHED(); asm volatile("s_nop 7\n\ts_nop 7" ::: "memory"); SCHED();
                             send_a(I0, I1); send_a(I0, I2); send_a(I0, I3);
                         } else if (u == UF) finish_ph1();
                         else finish_b(0, LDS_STG0);
                     }
                 },
                 [&]() __attribute__((always_inline)) {
                     if (__builtin_expect((maskB >> u) & 1u, 0)) finish_b(0, LDS_STG0);
                 });
        }
        SCHED(); asm volatile("s_nop 7\n\ts_nop 7" ::: "memory"); SCHED();
        send_a(I1, I1); send_a(I1, I2); send_a(I1, I3);
        {
            const unsigned o = ldsb + LDS_OWN + w * 4096 + l16;
            asm volatile("ds_write_b128 %0, a[64:67]\n\tds_write_b128 %0, a[68:71] offset:1024\n\tds_write_b128 %0, a[72:75] offset:2048\n\tds_write_b128 %0, a[76:79] offset:3072" :: "v"(o) : "memory");
        }
        wait_lds();
        bar();
        bar();
        zero_slots(0);
        wait_vm<0>();
    }
    bar();
    if (wid == 0) {
        const int src = p.rank;
        gu32* cnt = (gu32*)(myws + WS_CNT + (size_t)b * 128);
        if (lane == 0) {
            gu32* flag = (gu32*)(p.ws[dst] + WS_FLAGS + (size_t)(src * 32 + kq * 8 + mi * 2 + sub) * 128);
            __hip_atomic_store(flag, call, __ATOMIC_RELAXED, __HIP_MEMORY_SCOPE_SYSTEM);
            __hip_atomic_store(cnt, call, __ATOMIC_RELAXED, __HIP_MEMORY_SCOPE_AGENT);
            gu32* mine = (gu32*)(myws + WS_FLAGS + (size_t)b * 128);
            const unsigned long long t0 = __builtin_amdgcn_s_memrealtime();
            while ((int)(__hip_atomic_load(mine, __ATOMIC_RELAXED, __HIP_MEMORY_SCOPE_SYSTEM) - call) < 0) {
                if (give_up(t0)) {
                    gu16* o = (gu16*)(p.out[p.rank] + (size_t)p.rank * MLOC * NCOL * 2);
                    __hip_atomic_store(o, (unsigned short)0x7fc0, __ATOMIC_RELAXED, __HIP_MEMORY_SCOPE_SYSTEM);
                    break;
                }
                __builtin_amdgcn_s_sleep(1);
            }
        }
    }
}

}

extern "C" void kernel_launch(int rank, int nranks, ncclComm_t comm, hipStream_t stream,
                              void* const* d_in, const int* in_sizes, int n_in,
                              void* d_out, int out_size, const int* in_dtypes, int out_dtype,
                              void* const* d_out_all, void* const* d_ws_all, size_t ws_bytes) {
    (void)comm; (void)in_sizes; (void)n_in; (void)d_out; (void)out_size; (void)in_dtypes; (void)out_dtype;
    if (nranks != NR || rank < 0 || rank >= NR || ws_bytes < WS_END) { fprintf(stderr, "unsupported configuration\n"); return; }
    static bool attr = false;
    if (!attr) {
        HIP_CHECK(hipFuncSetAttribute((const void*)fused_kernel, hipFuncAttributeMaxDynamicSharedMemorySize, LDS_BYTES));
        attr = true;
    }
    Params p;
    p.x = (const char*)d_in[0];
    p.w = (const char*)d_in[1];
    for (int i = 0; i < NR; ++i) { p.out[i] = (char*)d_out_all[i]; p.ws[i] = (char*)d_ws_all[i]; }
    p.rank = rank;
    p.pad = 0;
    hipLaunchKernelGGL(fused_kernel, dim3(NWG), dim3(512), LDS_BYTES, stream, p);
    HIP_CHECK(hipGetLastError());
}
```

```cpp
#include <hip/hip_runtime.h>
#include <rccl/rccl.h>
#include <cstdio>
#include <type_traits>

#define HIP_CHECK(expr) do { hipError_t _e = (expr); if (_e != hipSuccess) { \
    fprintf(stderr, "HIP error %s at %s:%d\n", hipGetErrorString(_e), __FILE__, __LINE__); \
    return; } } while (0)

namespace {

#define LAS __attribute__((address_space(3)))
#define GAS __attribute__((address_space(1)))
typedef short bf16x8 __attribute__((ext_vector_type(8)));
typedef short bf16x4 __attribute__((ext_vector_type(4)));
typedef float f32x4 __attribute__((ext_vector_type(4)));
typedef unsigned u32x4 __attribute__((ext_vector_type(4)));
typedef unsigned u32x2 __attribute__((ext_vector_type(2)));
typedef GAS unsigned gu32;
typedef GAS unsigned short gu16;

constexpr int NR = 8;
constexpr int MLOC = 512;
constexpr int KD = 4096, ND = 4096;
constexpr int NCOL = ND / NR;
constexpr int BM = 128, BN = 128, BK = 64;
constexpr int SPP = 16;
constexpr int NSTEP = 2 * SPP;
constexpr int SLOT = 16384;
constexpr int NS = 4;
constexpr int LDS_B = NS * SLOT;
constexpr int LDS_XB = 2 * NS * SLOT;
constexpr int LDS_STG0 = LDS_XB + 3 * 8192;
constexpr int LDS_OWN = 0;
constexpr int LDS_STG1 = 16384;
constexpr int LDS_BYTES = LDS_STG0 + 8192;
constexpr int NWG = 256;
constexpr int LPT = 8;
constexpr int XT = 19;
constexpr int NCLS = 8;
constexpr int TSLEEP = 4;
constexpr int LEAD_BS = 1;

constexpr size_t WS_FLAGS = 0;
constexpr size_t WS_CNT = WS_FLAGS + (size_t)NWG * 128;
constexpr size_t WS_TMO = WS_CNT + (size_t)NWG * 128;
constexpr size_t WS_INBOX = 1u << 20;
constexpr size_t INBOX_TILE = (size_t)4 * 3 * 4 * 2 * 1024;
constexpr size_t WS_INBOX_BYTES = 2 * 64 * INBOX_TILE;
constexpr size_t WS_END = WS_INBOX + WS_INBOX_BYTES;
static_assert(WS_TMO + 128 <= WS_INBOX, "layout");
constexpr unsigned long long DEADLINE_TICKS = 20000000ull;

struct Params {
    const char* x;
    const char* w;
    char* out[NR];
    char* ws[NR];
    char* myws;
    int rank;
    int pad;
};

template <int N> __device__ __forceinline__ void wait_vm() { asm volatile("s_waitcnt vmcnt(%0)" :: "n"(N) : "memory"); }
__device__ __forceinline__ void wait_lds() { asm volatile("s_waitcnt lgkmcnt(0)" ::: "memory"); }
__device__ __forceinline__ void bar() { asm volatile("" ::: "memory"); __builtin_amdgcn_s_barrier(); asm volatile("" ::: "memory"); }
#define SCHED() __builtin_amdgcn_sched_barrier(0)
#define ACLOB0 "a0","a1","a2","a3","a4","a5","a6","a7","a8","a9","a10","a11","a12","a13","a14","a15"
#define ACLOB1 "a16","a17","a18","a19","a20","a21","a22","a23","a24","a25","a26","a27","a28","a29","a30","a31"
#define ACLOB2 "a32","a33","a34","a35","a36","a37","a38","a39","a40","a41","a42","a43","a44","a45","a46","a47"
#define ACLOB3 "a48","a49","a50","a51","a52","a53","a54","a55","a56","a57","a58","a59","a60","a61","a62","a63"
#define ACLOB4 "a64","a65","a66","a67","a68","a69","a70","a71","a72","a73","a74","a75","a76","a77","a78","a79"
#define ACLOB5 "a80","a81","a82","a83","a84","a85","a86","a87","a88","a89","a90","a91","a92","a93","a94","a95"
#define ACLOB6 "a96","a97","a98","a99","a100","a101","a102","a103","a104","a105","a106","a107","a108","a109","a110","a111"
#define ACLOB7 "a112","a113","a114","a115","a116","a117","a118","a119","a120","a121","a122","a123","a124","a125","a126","a127"
template <int R> __device__ __forceinline__ void areg_read16(float (&v)[16]) {
    asm volatile("s_nop 1\n\tv_accvgpr_read_b32 %0, a%c16\n\tv_accvgpr_read_b32 %1, a%c17\n\tv_accvgpr_read_b32 %2, a%c18\n\tv_accvgpr_read_b32 %3, a%c19\n\t"
                 "v_accvgpr_read_b32 %4, a%c20\n\tv_accvgpr_read_b32 %5, a%c21\n\tv_accvgpr_read_b32 %6, a%c22\n\tv_accvgpr_read_b32 %7, a%c23\n\t"
                 "v_accvgpr_read_b32 %8, a%c24\n\tv_accvgpr_read_b32 %9, a%c25\n\tv_accvgpr_read_b32 %10, a%c26\n\tv_accvgpr_read_b32 %11, a%c27\n\t"
                 "v_accvgpr_read_b32 %12, a%c28\n\tv_accvgpr_read_b32 %13, a%c29\n\tv_accvgpr_read_b32 %14, a%c30\n\tv_accvgpr_read_b32 %15, a%c31"
                 : "=v"(v[0]), "=v"(v[1]), "=v"(v[2]), "=v"(v[3]), "=v"(v[4]), "=v"(v[5]), "=v"(v[6]), "=v"(v[7]),
                   "=v"(v[8]), "=v"(v[9]), "=v"(v[10]), "=v"(v[11]), "=v"(v[12]), "=v"(v[13]), "=v"(v[14]), "=v"(v[15])
                 : "i"(R), "i"(R + 1), "i"(R + 2), "i"(R + 3), "i"(R + 4), "i"(R + 5), "i"(R + 6), "i"(R + 7),
                   "i"(R + 8), "i"(R + 9), "i"(R + 10), "i"(R + 11), "i"(R + 12), "i"(R + 13), "i"(R + 14), "i"(R + 15));
}

__device__ __forceinline__ float silu(float y) {
    const float c = __builtin_amdgcn_fmed3f(y, -60.0f, 60.0f);
    const float e = __builtin_amdgcn_exp2f(c * -1.44269504088896340736f);
    return y * __builtin_amdgcn_rcpf(1.0f + e);
}
__device__ __forceinline__ unsigned cvt_pk(float lo, float hi) {
    unsigned r; asm("v_cvt_pk_bf16_f32 %0, %1, %2" : "=v"(r) : "v"(lo), "v"(hi)); return r;
}
__device__ __forceinline__ __amdgpu_buffer_rsrc_t make_rsrc(const char* base, unsigned bytes) {
    return __builtin_amdgcn_make_buffer_rsrc((void*)base, 0, bytes, 0x00020000);
}
constexpr int AUX_SYS = 17;

struct F { bf16x8 a[2][4]; bf16x4 b[2][2]; };

__device__ __forceinline__ int korder(int tt, int g) {
    constexpr int bs = LEAD_BS, nb = SPP / bs;
    const int tau = tt / bs, s = tt % bs;
    int u = tau <= g ? tau : tau + 3 - g;
    if (u >= nb) u += g + 1;
    return (g + 4 * (u >> 2) - (u & 3)) * bs + s;
}

template <int N> __device__ __forceinline__ bool valid(const u32x4 (&r)[N]) {
    unsigned m = ~0u;
#pragma unroll
    for (int k = 0; k < N; ++k) { m = min(m, min(r[k].x, r[k].y)); m = min(m, min(r[k].z, r[k].w)); }
    return __all(m != 0u);
}
__device__ __forceinline__ void add_bf16x16(float (&v)[16], const u32x4& lo, const u32x4& hi) {
    const unsigned wd[8] = {lo.x, lo.y, lo.z, lo.w, hi.x, hi.y, hi.z, hi.w};
#pragma unroll
    for (int i = 0; i < 8; ++i) {
        v[2 * i] += __builtin_bit_cast(float, wd[i] << 16);
        v[2 * i + 1] += __builtin_bit_cast(float, wd[i] & 0xffff0000u);
    }
}

__global__ __launch_bounds__(512, 2) void fused_kernel(Params p) {
    extern __shared__ __attribute__((aligned(16))) unsigned char shm[];
    LAS unsigned char* lds = (LAS unsigned char*)shm;
    const int tid = threadIdx.x;
    const int wid = __builtin_amdgcn_readfirstlane(tid >> 6);
    const int lane = tid & 63;
    const int b = blockIdx.x;
    const int kq = b & 3, hh = (b >> 2) & 1, idx = b >> 3;
    const int mi = idx & 3, nl = idx >> 2;
    const int dst = nl, sub = hh;
    const int T = hh * 32 + idx;
    const int m0 = mi * BM;
    const int w = wid & 3;
    const int l31 = lane & 31, h = lane >> 5;
    const unsigned ldsb = (unsigned)(size_t)lds;
    const unsigned l16 = lane * 16;
    char* myws = p.myws;
    const char* const px = p.x;
    const char* const pw = p.w;
    char* const pout = p.out[dst];
    asm volatile("" :: "s"(px), "s"(pw), "s"(myws), "s"(pout));
    unsigned call = 0;
    bool bad = false;

    auto inbox = [&](int ph, int q, int j) __attribute__((always_inline)) -> unsigned {
        return (unsigned)(WS_INBOX + (size_t)(ph * 64 + T) * INBOX_TILE + (size_t)((q * 3 + (j - 1)) * 4 + w) * 2048);
    };
    auto give_up = [&](unsigned long long t0) __attribute__((always_inline)) -> bool {
        gu32* tmo = (gu32*)(myws + WS_TMO);
        if (__hip_atomic_load(tmo, __ATOMIC_RELAXED, __HIP_MEMORY_SCOPE_AGENT) == call
            || __builtin_amdgcn_s_memrealtime() - t0 > DEADLINE_TICKS) {
            __hip_atomic_store(tmo, call, __ATOMIC_RELAXED, __HIP_MEMORY_SCOPE_AGENT);
            return true;
        }
        return false;
    };
    auto finish_a = [&](int stg_off, const float (&v)[16]) __attribute__((always_inline)) {
        LAS unsigned char* stg = lds + stg_off + l31 * 256 + h * 8;
        unsigned o[8];
#pragma unroll
        for (int i = 0; i < 8; ++i) o[i] = cvt_pk(silu(v[2 * i]), silu(v[2 * i + 1]));
        if (__builtin_expect(bad, 0)) {
#pragma unroll
            for (int i = 0; i < 8; ++i) o[i] = 0x7fc07fc0u;
        }
#pragma unroll
        for (int e4 = 0; e4 < 4; ++e4) *(LAS u32x2*)(stg + (((w * 4 + e4) ^ (l31 & 15)) << 4)) = (u32x2){o[2 * e4], o[2 * e4 + 1]};
        wait_lds();
    };
    const __amdgpu_buffer_rsrc_t rs_out = make_rsrc(pout, (unsigned)(ND * NCOL * 2));
    auto finish_b = [&](int ph, int stg_off) __attribute__((always_inline)) {
        const unsigned obase = (unsigned)(((p.rank * MLOC + m0 + kq * 32) * NCOL + ph * 256 + sub * 128) * 2);
#pragma unroll
        for (int i = 0; i < 2; ++i) {
            const int r = (w * 2 + i) * 4 + (lane >> 4), c = lane & 15;
            const u32x4 v = *(LAS u32x4*)(lds + stg_off + r * 256 + ((c ^ (r & 15)) << 4));
            __builtin_amdgcn_raw_buffer_store_b128(v, rs_out, r * (NCOL * 2) + c * 16, obase, AUX_SYS);
        }
    };
    const __amdgpu_buffer_rsrc_t rs_ws = make_rsrc(myws, (unsigned)WS_END);
    auto zero_slots = [&](int ph) __attribute__((always_inline)) {
#pragma unroll
        for (int k = 0; k < 6; ++k)
            __builtin_amdgcn_raw_buffer_store_b128((u32x4){0u, 0u, 0u, 0u}, rs_ws, l16 + (k & 1) * 1024, inbox(ph, kq, 1 + (k >> 1)), AUX_SYS);
    };
    const gu32* cnt_ptr = (const gu32*)(myws + WS_CNT + (size_t)b * 128);

    if (__builtin_expect(wid >= 4, 1)) {
        unsigned callv;
        unsigned offA[4], offB[4];
#pragma unroll
        for (int i = 0; i < 4; ++i) {
            const int row = (w * 4 + i) * 8 + (lane >> 3);
            const int ch = (lane & 7) ^ ((row >> 1) & 7);
            offA[i] = (unsigned)(row * (KD * 2) + ch * 16);
            const int k = (w * 4 + i) * 4 + (lane >> 4);
            const int cb = (lane & 15) ^ ((k & 3) << 2);
            offB[i] = (unsigned)(k * (ND * 2) + cb * 16);
        }
        const char* gA = px + (size_t)m0 * (KD * 2) + (size_t)kq * (SPP * BK * 2);
        const char* gB = pw + (size_t)kq * ((size_t)SPP * BK * ND * 2) + (size_t)(dst * NCOL + sub * BN) * 2;
        const int g = (mi + nl) & 3;
        struct Src { const char* a; const char* b; };
        auto source = [&](int t) __attribute__((always_inline)) -> Src {
            const int kt = korder(t & (SPP - 1), g);
            return Src{gA + kt * (BK * 2), gB + (size_t)kt * ((size_t)BK * ND * 2) + (t >> 4) * (256 * 2)};
        };
        auto issue = [&](const Src& q, int slot) __attribute__((always_inline)) {
            const unsigned la = ldsb + slot * SLOT + w * 4096;
#pragma unroll
            for (int i = 0; i < 4; ++i)
                asm volatile("s_mov_b32 m0, %2\n\ts_nop 0\n\tglobal_load_lds_dwordx4 %0, %1" :: "v"(offA[i]), "s"(q.a), "s"(la + i * 1024) : "memory");
#pragma unroll
            for (int i = 0; i < 4; ++i)
                asm volatile("s_mov_b32 m0, %2\n\ts_nop 0\n\tglobal_load_lds_dwordx4 %0, %1" :: "v"(offB[i]), "s"(q.b), "s"(la + LDS_B + i * 1024) : "memory");
        };
        auto issue_x = [&](int j) __attribute__((always_inline)) {
            const char* src = myws + inbox(0, kq, j);
#pragma unroll
            for (int i = 0; i < 2; ++i)
                asm volatile("s_mov_b32 m0, %2\n\ts_nop 0\n\tglobal_load_lds_dwordx4 %0, %1 sc0 sc1" :: "v"(l16 + i * 1024), "s"(src), "s"(ldsb + LDS_XB + (j - 1) * 8192 + w * 2048 + i * 1024) : "memory");
        };
        issue(source(0), 0);
        issue(source(1), 1);
        asm volatile("global_load_dword %0, %1, off sc1" : "=v"(callv) : "v"(cnt_ptr) : "memory");
        Src nx = source(2);
        auto step = [&](int t) __attribute__((always_inline)) {
            issue(nx, (t + 3) & (NS - 1));
            nx = source((t + 4) & (NSTEP - 1));
        };
#pragma unroll 1
        for (int t = -1; t < XT; ++t) { wait_vm<LPT>(); bar(); step(t); }
        wait_vm<LPT>(); bar();
        issue_x(1); issue_x(2); issue_x(3);
        step(XT);
        wait_vm<LPT + 6>(); bar();
        step(XT + 1);
#pragma unroll 1
        for (int t = XT + 2; t <= NSTEP - 4; ++t) { wait_vm<LPT>(); bar(); step(t); }
        wait_vm<LPT>(); bar();
        wait_vm<0>(); bar();
        bar();
        bar();
        asm volatile("" : "+v"(callv));
        call = __builtin_amdgcn_readfirstlane(callv) + 1;
        u32x4 in[6];
        {
            const unsigned long long t0 = __builtin_amdgcn_s_memrealtime();
            while (true) {
#pragma unroll
                for (int k = 0; k < 6; ++k) in[k] = __builtin_amdgcn_raw_buffer_load_b128(rs_ws, l16 + (k & 1) * 1024, inbox(1, kq, 1 + (k >> 1)), AUX_SYS);
                if (valid<6>(in)) break;
                if (give_up(t0)) { bad = true; break; }
                __builtin_amdgcn_s_sleep(TSLEEP);
            }
        }
        float v[16];
#pragma unroll
        for (int e4 = 0; e4 < 4; ++e4) {
            const f32x4 own = *(LAS f32x4*)(lds + LDS_OWN + w * 4096 + e4 * 1024 + l16);
#pragma unroll
            for (int e = 0; e < 4; ++e) v[4 * e4 + e] = own[e];
        }
        add_bf16x16(v, in[0], in[1]); add_bf16x16(v, in[2], in[3]); add_bf16x16(v, in[4], in[5]);
        finish_a(LDS_STG1, v);
        bar();
        finish_b(1, LDS_STG1);
        wait_vm<0>();
    } else {
        __builtin_amdgcn_s_setprio(2);
        call = __hip_atomic_load(cnt_ptr, __ATOMIC_RELAXED, __HIP_MEMORY_SCOPE_AGENT) + 1;
        unsigned xa[4][4];
#pragma unroll
        for (int j = 0; j < 4; ++j)
#pragma unroll
            for (int ks = 0; ks < 4; ++ks) xa[j][ks] = ldsb + ((j ^ kq) * 32 + l31) * 128 + (((2 * ks + h) ^ ((l31 >> 1) & 7)) << 4);
        const int blk = (lane >> 4) & 1, q4 = (lane & 15) >> 2, p4 = lane & 3;
        unsigned wa = ldsb + LDS_B + (8 * h + q4) * 256 + (((2 * w + blk) ^ (q4 << 1)) << 5) + 8 * p4;
        auto rd1 = [&](F& f, int so, int hf, int r) __attribute__((always_inline)) {
            const int s = r / 6, rr = r % 6, ks = 2 * hf + s;
            if (rr < 2) f.b[s][rr] = __builtin_amdgcn_ds_read_tr16_b64_v4i16((LAS bf16x4*)(size_t)(wa + so + ks * 4096 + rr * 1024));
            else f.a[s][rr - 2] = *(LAS bf16x8*)(size_t)(xa[rr - 2][ks] + so);
        };
        auto half = [&](bool ph1, const F& cur, F& nxt, int so, int hf) __attribute__((always_inline)) {
#pragma unroll
            for (int s = 0; s < 2; ++s) {
                const bf16x8 wv = __builtin_shufflevector(cur.b[s][0], cur.b[s][1], 0, 1, 2, 3, 4, 5, 6, 7);
#pragma unroll
                for (int j = 0; j < 4; ++j) {
#define MFMA_A(R, CL) asm volatile("v_mfma_f32_32x32x16_bf16 " R ", %0, %1, " R :: "v"(wv), "v"(cur.a[s][j]) : CL)
                    if (ph1) {
                        if (j == 0) MFMA_A("a[0:15]", ACLOB0); else if (j == 1) MFMA_A("a[16:31]", ACLOB1);
                        else if (j == 2) MFMA_A("a[32:47]", ACLOB2); else MFMA_A("a[48:63]", ACLOB3);
                    } else {
                        if (j == 0) MFMA_A("a[64:79]", ACLOB4); else if (j == 1) MFMA_A("a[80:95]", ACLOB5);
                        else if (j == 2) MFMA_A("a[96:111]", ACLOB6); else MFMA_A("a[112:127]", ACLOB7);
                    }
#undef MFMA_A
                    SCHED();
                    const int i = s * 4 + j;
                    const int r0 = i < 4 ? 2 * i : 4 + i, n = i < 4 ? 2 : 1;
                    rd1(nxt, so, hf, r0); if (n == 2) rd1(nxt, so, hf, r0 + 1);
                    SCHED();
                }
            }
        };
        auto toggle = [&]() __attribute__((always_inline)) {
#pragma unroll
            for (int j = 0; j < 4; ++j)
#pragma unroll
                for (int ks = 0; ks < 4; ++ks) xa[j][ks] ^= 2 * SLOT;
            wa ^= 2 * SLOT;
        };
        F f0, f1;
        auto pair = [&](bool ph1, auto&& actA, auto&& actB) __attribute__((always_inline)) {
            bar();
            actA(); SCHED();
            half(ph1, f0, f1, 0, 1);
            half(ph1, f1, f0, SLOT, 0);
            bar();
            actB(); SCHED();
            half(ph1, f0, f1, SLOT, 1);
            toggle();
            SCHED();
            half(ph1, f1, f0, 0, 0);
        };
        auto none = [&]() __attribute__((always_inline)) {};
        auto send_a = [&](auto pc, auto jc) __attribute__((always_inline)) {
            constexpr int ph = decltype(pc)::value, j = decltype(jc)::value, R = 64 * ph + 16 * j;
            const unsigned off = inbox(ph, kq ^ j, j);
            float v[16];
            areg_read16<R>(v);
            unsigned wd[8];
#pragma unroll
            for (int i = 0; i < 8; ++i) wd[i] = cvt_pk(v[2 * i], v[2 * i + 1]);
            const unsigned m = min(min(min(wd[0], wd[1]), min(wd[2], wd[3])), min(min(wd[4], wd[5]), min(wd[6], wd[7])));
            if (__builtin_expect(__any(m == 0u), 0)) {
#pragma unroll
                for (int i = 0; i < 8; ++i) wd[i] |= (wd[i] - 1u) & 0x80000000u;
            }
            __builtin_amdgcn_raw_buffer_store_b128((u32x4){wd[0], wd[1], wd[2], wd[3]}, rs_ws, l16, off, AUX_SYS);
            __builtin_amdgcn_raw_buffer_store_b128((u32x4){wd[4], wd[5], wd[6], wd[7]}, rs_ws, l16 + 1024, off, AUX_SYS);
        };
        constexpr std::integral_constant<int, 0> I0{}; constexpr std::integral_constant<int, 1> I1{};
        constexpr std::integral_constant<int, 2> I2{}; constexpr std::integral_constant<int, 3> I3{};
        auto finish_ph1 = [&]() __attribute__((always_inline)) {
            u32x4 r[6];
#pragma unroll
            for (int k = 0; k < 6; ++k) r[k] = *(LAS u32x4*)(lds + LDS_XB + (k >> 1) * 8192 + w * 2048 + (k & 1) * 1024 + l16);
            float v[16];
            areg_read16<0>(v);
            if (__builtin_expect(!valid<6>(r), 0)) {
                const unsigned long long t0 = __builtin_amdgcn_s_memrealtime();
                while (true) {
#pragma unroll
                    for (int k = 0; k < 6; ++k) r[k] = __builtin_amdgcn_raw_buffer_load_b128(rs_ws, l16 + (k & 1) * 1024, inbox(0, kq, 1 + (k >> 1)), AUX_SYS);
                    if (valid<6>(r)) break;
                    if (give_up(t0)) { bad = true; break; }
                }
            }
            add_bf16x16(v, r[0], r[1]); add_bf16x16(v, r[2], r[3]); add_bf16x16(v, r[4], r[5]);
            finish_a(LDS_STG0, v);
        };

        asm volatile(
#define Z4(a, b, c, d) "v_accvgpr_write_b32 a" #a ", 0\n\tv_accvgpr_write_b32 a" #b ", 0\n\tv_accvgpr_write_b32 a" #c ", 0\n\tv_accvgpr_write_b32 a" #d ", 0\n\t"
            Z4(0,1,2,3) Z4(4,5,6,7) Z4(8,9,10,11) Z4(12,13,14,15) Z4(16,17,18,19) Z4(20,21,22,23) Z4(24,25,26,27) Z4(28,29,30,31) Z4(32,33,34,35) Z4(36,37,38,39) Z4(40,41,42,43) Z4(44,45,46,47) Z4(48,49,50,51) Z4(52,53,54,55) Z4(56,57,58,59) Z4(60,61,62,63) Z4(64,65,66,67) Z4(68,69,70,71) Z4(72,73,74,75) Z4(76,77,78,79) Z4(80,81,82,83) Z4(84,85,86,87) Z4(88,89,90,91) Z4(92,93,94,95) Z4(96,97,98,99) Z4(100,101,102,103) Z4(104,105,106,107) Z4(108,109,110,111) Z4(112,113,114,115) Z4(116,117,118,119) Z4(120,121,122,123) Z4(124,125,126,127)
#undef Z4
            "s_nop 1" ::: ACLOB0, ACLOB1, ACLOB2, ACLOB3, ACLOB4, ACLOB5, ACLOB6, ACLOB7);
        bar();
#pragma unroll
        for (int r = 0; r < 12; ++r) rd1(f0, 0, 0, r);
#pragma unroll 1
        for (int u = 0; u < SPP / 2; ++u) {
            pair(true, none, none);
        }
        constexpr int UF = (XT + 3 - 16) / 2;
        const int cls0 = NCLS == 8 ? mi + 4 * (w & 1) : mi;
        const int cls = cls0 > 27 - XT ? 27 - XT : cls0;
        const int sstep = XT + 4 + cls - 16;
        unsigned maskA = 1u | (1u << UF) | ((sstep & 1) ? 0u : 1u << (sstep >> 1)), maskB = (sstep & 1) ? 1u << (sstep >> 1) : 0u;
        asm volatile("" : "+s"(maskA), "+s"(maskB));
#pragma unroll 1
        for (int u = 0; u < SPP / 2; ++u) {
            pair(false,
                 [&]() __attribute__((always_inline)) {
                     if (__builtin_expect((maskA >> u) & 1u, 0)) {
                         if (u == 0) {
                             SCHED(); asm volatile("s_nop 7\n\ts_nop 7" ::: "memory"); SCHED();
                             send_a(I0, I1); send_a(I0, I2); send_a(I0, I3);
                         } else if (u == UF) finish_ph1();
                         else finish_b(0, LDS_STG0);
                     }
                 },
                 [&]() __attribute__((always_inline)) {
                     if (__builtin_expect((maskB >> u) & 1u, 0)) finish_b(0, LDS_STG0);
                 });
        }
        SCHED(); asm volatile("s_nop 7\n\ts_nop 7" ::: "memory"); SCHED();
        send_a(I1, I1); send_a(I1, I2); send_a(I1, I3);
        {
            const unsigned o = ldsb + LDS_OWN + w * 4096 + l16;
            asm volatile("ds_write_b128 %0, a[64:67]\n\tds_write_b128 %0, a[68:71] offset:1024\n\tds_write_b128 %0, a[72:75] offset:2048\n\tds_write_b128 %0, a[76:79] offset:3072" :: "v"(o) : "memory");
        }
        wait_lds();
        bar();
        bar();
        wait_vm<0>();
    }
    bar();
    if (wid != 0) zero_slots(wid >= 4 ? 1 : 0);
    if (wid == 0) {
        const int src = p.rank;
        gu32* cnt = (gu32*)(myws + WS_CNT + (size_t)b * 128);
        if (lane == 0) {
            gu32* flag = (gu32*)(p.ws[dst] + WS_FLAGS + (size_t)(src * 32 + kq * 8 + mi * 2 + sub) * 128);
            __hip_atomic_store(flag, call, __ATOMIC_RELAXED, __HIP_MEMORY_SCOPE_SYSTEM);
            __hip_atomic_store(cnt, call, __ATOMIC_RELAXED, __HIP_MEMORY_SCOPE_AGENT);
        }
        zero_slots(0);
        if (lane == 0) {
            gu32* mine = (gu32*)(myws + WS_FLAGS + (size_t)b * 128);
            const unsigned long long t0 = __builtin_amdgcn_s_memrealtime();
            while ((int)(__hip_atomic_load(mine, __ATOMIC_RELAXED, __HIP_MEMORY_SCOPE_SYSTEM) - call) < 0) {
                if (give_up(t0)) {
                    gu16* o = (gu16*)(p.out[p.rank] + (size_t)p.rank * MLOC * NCOL * 2);
                    __hip_atomic_store(o, (unsigned short)0x7fc0, __ATOMIC_RELAXED, __HIP_MEMORY_SCOPE_SYSTEM);
                    break;
                }
                __builtin_amdgcn_s_sleep(1);
            }
        }
    }
}

}

extern "C" void kernel_launch(int rank, int nranks, ncclComm_t comm, hipStream_t stream,
                              void* const* d_in, const int* in_sizes, int n_in,
                              void* d_out, int out_size, const int* in_dtypes, int out_dtype,
                              void* const* d_out_all, void* const* d_ws_all, size_t ws_bytes) {
    (void)comm; (void)in_sizes; (void)n_in; (void)d_out; (void)out_size; (void)in_dtypes; (void)out_dtype;
    if (nranks != NR || rank < 0 || rank >= NR || ws_bytes < WS_END) { fprintf(stderr, "unsupported configuration\n"); return; }
    static bool attr = false;
    if (!attr) {
        HIP_CHECK(hipFuncSetAttribute((const void*)fused_kernel, hipFuncAttributeMaxDynamicSharedMemorySize, LDS_BYTES));
        attr = true;
    }
    Params p;
    p.x = (const char*)d_in[0];
    p.w = (const char*)d_in[1];
    for (int i = 0; i < NR; ++i) { p.out[i] = (char*)d_out_all[i]; p.ws[i] = (char*)d_ws_all[i]; }
    p.myws = (char*)d_ws_all[rank];
    p.rank = rank;
    p.pad = 0;
    hipLaunchKernelGGL(fused_kernel, dim3(NWG), dim3(512), LDS_BYTES, stream, p);
    HIP_CHECK(hipGetLastError());
}
```
